```python
import numpy as np

kernel_name = "gpu_gemm_ar_m4096_k4096_n4096_e5m2e5m2_dqsilu"


ABSMAX_RTOL = 0.05

COMPUTE_DTYPE = "float32"
EPILOGUE = "dequant_silu"
IN_QUANT = None
OUT_QUANT = None
IN_FP8 = "e5m2e5m2"
IN_DTYPES = {"x": "e5m2", "w_mat": "e5m2",
             "scale_x": "float32", "scale_w": "float32"}
WORLD_SIZE = 8

def _snap_fp8(x, mant_bits, max_abs, min_normal):
    a = np.abs(x.astype(np.float64))
    _m, _e = np.frexp(a)
    q = np.where(a >= min_normal, np.ldexp(1.0, _e - 1 - mant_bits),
                 min_normal * 2.0 ** -mant_bits)
    snapped = np.minimum(np.round(a / q) * q, max_abs)
    return (np.sign(x) * snapped).astype(np.float32)

def setup_inputs():

    x = _snap_fp8(np.random.randn(4096, 4096).astype(np.float32), 2, 57344.0, 2.0 ** -14)
    w_mat = _snap_fp8(np.random.randn(4096, 4096).astype(np.float32), 2, 57344.0, 2.0 ** -14)
    scale_x = (0.02 * (1.0 + np.random.rand(1))).astype(np.float32)
    scale_w = (0.02 * (1.0 + np.random.rand(1))).astype(np.float32)
    return {"x": x, "w_mat": w_mat, "scale_x": scale_x, "scale_w": scale_w}

def reference(x, w_mat, scale_x, scale_w):
    acc = (x @ w_mat).astype(np.float32)
    y = (acc * (scale_x * scale_w)).astype(np.float32)
    return (y / (1.0 + np.exp(-np.clip(y, -60.0, 60.0)))).astype(np.float32)
```

```cpp
#include <hip/hip_runtime.h>
#include <rccl/rccl.h>
#include <cstdio>

#define HIP_CHECK(expr) do { hipError_t _e = (expr); if (_e != hipSuccess) { \
    fprintf(stderr, "HIP error %s at %s:%d\n", hipGetErrorString(_e), __FILE__, __LINE__); \
    return; } } while (0)

typedef unsigned int u32;
typedef unsigned char u8;
typedef __attribute__((ext_vector_type(4))) u32 u32x4;
typedef __attribute__((ext_vector_type(2))) u32 u32x2;
typedef __attribute__((ext_vector_type(8))) int i32x8;
typedef __attribute__((ext_vector_type(4))) float f32x4;

constexpr int MM = 4096, NN = 4096, NRANK = 8, KSH = 512;
constexpr int CK = 64;
constexpr int NCH = KSH / CK;
constexpr int NSTEP = NCH * 4;
constexpr int TM = 288, TN = 256, NRB = 15, NCB = 16;
constexpr int NPUSH = 16, GRID = NPUSH + NRB * NCB;
constexpr size_t OFF_WGCNT = 0;
constexpr size_t OFF_TMO = 32u << 10;
constexpr size_t OFF_FLAG = 64u << 10;
constexpr size_t OFF_STAGE = 1u << 20;
constexpr size_t MAT_BYTES = 16u << 20;
constexpr size_t WS_NEED = OFF_STAGE + 4 * MAT_BYTES;
constexpr unsigned long long DEADLINE_TICKS = 50000000ull;
constexpr int STAGE_BYTES = 68 * 1024;
constexpr int LDS_BYTES = 2 * STAGE_BYTES + 64;

struct Params {
  const u8* x; const u8* w; const float* sx; const float* sw; float* out;
  u8* ws[NRANK];
  int rank;
};

#define WAIT_VM(n) asm volatile("s_waitcnt vmcnt(" #n ")" ::: "memory")

__device__ __forceinline__ void st16_sys(u8* p, u32x4 v) {
  asm volatile("global_store_dwordx4 %0, %1, off sc0 sc1\n\ts_nop 1" :: "v"(p), "v"(v) : "memory");
}

__device__ __forceinline__ void tr4(u32 d0, u32 d1, u32 d2, u32 d3, u32& o0, u32& o1, u32& o2, u32& o3) {
  u32 t0 = __builtin_amdgcn_perm(d1, d0, 0x05010400u), t1 = __builtin_amdgcn_perm(d1, d0, 0x07030602u);
  u32 t2 = __builtin_amdgcn_perm(d3, d2, 0x05010400u), t3 = __builtin_amdgcn_perm(d3, d2, 0x07030602u);
  o0 = __builtin_amdgcn_perm(t2, t0, 0x05040100u); o1 = __builtin_amdgcn_perm(t2, t0, 0x07060302u);
  o2 = __builtin_amdgcn_perm(t3, t1, 0x05040100u); o3 = __builtin_amdgcn_perm(t3, t1, 0x07060302u);
}

#define LD16(dst, addr) asm volatile("global_load_dwordx4 %0, %1, off" : "=v"(dst) : "v"(addr) : "memory")
#define LD4(dst, addr) asm volatile("global_load_dword %0, %1, off" : "=v"(dst) : "v"(addr) : "memory")

__device__ __forceinline__ void pusher_x(const Params& p, u32 call, int q) {
  const int wid = __builtin_amdgcn_readfirstlane(threadIdx.x >> 6), lane = threadIdx.x & 63;
  const size_t pbase = OFF_STAGE + (size_t)(call & 1) * 2 * MAT_BYTES + (size_t)p.rank * NCH * 256 * 1024 + lane * 16;
  u32* myflag = (u32*)(p.ws[lane & 7] + OFF_FLAG);
  const int nun = (q + 8 < NRB) ? 2 * NCH : NCH;
  const u8* xl = p.x + (size_t)(lane & 15) * KSH + (lane >> 4) * 16;
  u32x4 n0, n1, n2, c0, c1, c2;
  const int nper = nun / NCH;
  auto issue = [&](int k) {
    const int c = k / nper, rb = q + 8 * (k % nper);
    const int nblk = (rb == NRB - 1) ? 4 : 18;
    const int t0 = wid < nblk ? wid : 0, t1 = wid + 8 < nblk ? wid + 8 : 0, t2 = wid + 16 < nblk ? wid + 16 : 0;
    const u8* base = xl + (size_t)rb * TM * KSH + c * CK;
    LD16(n0, base + (size_t)t0 * 16 * KSH);
    LD16(n1, base + (size_t)t1 * 16 * KSH);
    LD16(n2, base + (size_t)t2 * 16 * KSH);
  };
  issue(0);
  WAIT_VM(0);
  asm volatile("" : "+v"(n0), "+v"(n1), "+v"(n2));
  int prevf = -1;
  for (int k = 0; k < nun; ++k) {
    const int c = k / nper, rb = q + 8 * (k % nper);
    const int nblk = (rb == NRB - 1) ? 4 : 18;
    const int np = (wid < nblk) + (wid + 8 < nblk) + (wid + 16 < nblk);
    c0 = n0; c1 = n1; c2 = n2;
    asm volatile("" : "+v"(c0), "+v"(c1), "+v"(c2));
    const size_t doff = pbase + ((size_t)c * 256 + rb * 18 + wid) * 1024;
    issue(k + 1 < nun ? k + 1 : k);
    if (np == 3) {
#pragma unroll
      for (int i = 1; i <= NRANK; ++i) { u8* d = p.ws[(p.rank + i) & 7] + doff; st16_sys(d, c0); st16_sys(d + 8192, c1); st16_sys(d + 16384, c2); }
      WAIT_VM(24);
    } else if (np == 2) {
#pragma unroll
      for (int i = 1; i <= NRANK; ++i) { u8* d = p.ws[(p.rank + i) & 7] + doff; st16_sys(d, c0); st16_sys(d + 8192, c1); }
      WAIT_VM(16);
    } else if (np == 1) {
#pragma unroll
      for (int i = 1; i <= NRANK; ++i) { u8* d = p.ws[(p.rank + i) & 7] + doff; st16_sys(d, c0); }
      WAIT_VM(8);
    } else {
      WAIT_VM(0);
    }
    asm volatile("" : "+v"(n0), "+v"(n1), "+v"(n2));
    __builtin_amdgcn_s_barrier();
    if (prevf >= 0 && threadIdx.x < NRANK) __hip_atomic_fetch_add(myflag + prevf * 32, 1u, __ATOMIC_RELAXED, __HIP_MEMORY_SCOPE_SYSTEM);
    prevf = c * 32 + 2 * rb + 1;
  }
  WAIT_VM(0);
  __builtin_amdgcn_s_barrier();
  if (threadIdx.x < NRANK) __hip_atomic_fetch_add(myflag + prevf * 32, 1u, __ATOMIC_RELAXED, __HIP_MEMORY_SCOPE_SYSTEM);
}

__device__ __forceinline__ void pusher_w(const Params& p, u8* lds, u32 call, int q) {
  const int wid = __builtin_amdgcn_readfirstlane(threadIdx.x >> 6), lane = threadIdx.x & 63;
  const size_t pbase = OFF_STAGE + (size_t)(call & 1) * 2 * MAT_BYTES + MAT_BYTES + (size_t)p.rank * NCH * 256 * 1024 + lane * 16;
  u32* myflag = (u32*)(p.ws[lane & 7] + OFF_FLAG);
  u8* wreg = lds + wid * 2048;
  const int lq = lane & 7, pc = (lane >> 3) & 3, hf = lane >> 5;
  const u8* wl = p.w + (size_t)(pc * 16 + hf * 8) * NN + wid * 32 + lq * 4;
  u8* wp = wreg + (lq >> 2) * 1024 + pc * 256 + (lq & 3) * 64 + hf * 8;
  u32 d0, d1, d2, d3, d4, d5, d6, d7;
  u32x4 c0, c1;
  constexpr int nun = 2 * NCH;
  auto issue = [&](int k) {
    const int c = k >> 1, cb = q + 8 * (k & 1);
    const u8* a = wl + (size_t)c * CK * NN + cb * TN;
    LD4(d0, a); LD4(d1, a + 1 * NN); LD4(d2, a + 2 * NN); LD4(d3, a + 3 * NN);
    LD4(d4, a + 4 * NN); LD4(d5, a + 5 * NN); LD4(d6, a + 6 * NN); LD4(d7, a + 7 * NN);
  };
  issue(0);
  WAIT_VM(0);
  asm volatile("" : "+v"(d0), "+v"(d1), "+v"(d2), "+v"(d3), "+v"(d4), "+v"(d5), "+v"(d6), "+v"(d7));
  int prevf = -1;
  for (int k = 0; k < nun; ++k) {
    const int c = k >> 1, cb = q + 8 * (k & 1);
    u32 a0, a1, a2, a3, b0, b1, b2, b3;
    tr4(d0, d1, d2, d3, a0, a1, a2, a3);
    tr4(d4, d5, d6, d7, b0, b1, b2, b3);
    *(u32x2*)(wp + 0) = u32x2{a0, b0};
    *(u32x2*)(wp + 16) = u32x2{a1, b1};
    *(u32x2*)(wp + 32) = u32x2{a2, b2};
    *(u32x2*)(wp + 48) = u32x2{a3, b3};
    asm volatile("s_waitcnt lgkmcnt(0)" ::: "memory");
    c0 = *(const u32x4*)(wreg + lane * 16);
    c1 = *(const u32x4*)(wreg + 1024 + lane * 16);
    asm volatile("s_waitcnt lgkmcnt(0)" : "+v"(c0), "+v"(c1) :: "memory");
    const size_t doff = pbase + ((size_t)c * 256 + cb * 16 + wid * 2) * 1024;
    issue(k + 1 < nun ? k + 1 : k);
#pragma unroll
    for (int i = 1; i <= NRANK; ++i) { u8* d = p.ws[(p.rank + i) & 7] + doff; st16_sys(d, c0); st16_sys(d + 1024, c1); }
    WAIT_VM(16);
    asm volatile("" : "+v"(d0), "+v"(d1), "+v"(d2), "+v"(d3), "+v"(d4), "+v"(d5), "+v"(d6), "+v"(d7));
    __builtin_amdgcn_s_barrier();
    if (prevf >= 0 && threadIdx.x < NRANK) __hip_atomic_fetch_add(myflag + prevf * 32, 1u, __ATOMIC_RELAXED, __HIP_MEMORY_SCOPE_SYSTEM);
    prevf = c * 32 + 2 * cb;
  }
  WAIT_VM(0);
  __builtin_amdgcn_s_barrier();
  if (threadIdx.x < NRANK) __hip_atomic_fetch_add(myflag + prevf * 32, 1u, __ATOMIC_RELAXED, __HIP_MEMORY_SCOPE_SYSTEM);
}

template <typename T> __device__ __forceinline__ T* uniform_ptr(T* q) {
  unsigned long long v = (unsigned long long)q;
  u32 lo = __builtin_amdgcn_readfirstlane((u32)v), hi = __builtin_amdgcn_readfirstlane((u32)(v >> 32));
  return (T*)(((unsigned long long)hi << 32) | lo);
}
__device__ __forceinline__ void glds2(const u8* gbase, u32 ldsaddr, u32 voff) {
  u32 keep;
  asm volatile("s_mov_b32 %0, m0\n\ts_mov_b32 m0, %2\n\ts_nop 0\n\t"
               "global_load_lds_dwordx4 %1, %3\n\t"
               "global_load_lds_dwordx4 %1, %3 offset:1024\n\t"
               "s_mov_b32 m0, %0"
               : "=&s"(keep) : "v"(voff), "s"(ldsaddr), "s"(gbase) : "memory");
}
__device__ __forceinline__ void glds1(const u8* gbase, u32 ldsaddr, u32 voff) {
  u32 keep;
  asm volatile("s_mov_b32 %0, m0\n\ts_mov_b32 m0, %2\n\ts_nop 0\n\t"
               "global_load_lds_dwordx4 %1, %3\n\t"
               "s_mov_b32 m0, %0"
               : "=&s"(keep) : "v"(voff), "s"(ldsaddr), "s"(gbase) : "memory");
}
__device__ __forceinline__ void gemm_stage(const u8* Ag, const u8* Bg, u32 dst, int s, int wid, u32 voff) {
  const int c = s >> 2, s0 = (s & 3) * 2;
  const size_t o0 = (size_t)(s0 * NCH + c) * (256 * 1024), o1 = o0 + (size_t)NCH * (256 * 1024);
  glds2(Ag + o0 + wid * 2048, dst + wid * 2048, voff);
  glds2(Ag + o1 + wid * 2048, dst + 18 * 1024 + wid * 2048, voff);
  glds2(Bg + o0 + wid * 2048, dst + 36 * 1024 + wid * 2048, voff);
  glds2(Bg + o1 + wid * 2048, dst + 52 * 1024 + wid * 2048, voff);
  if (wid < 2) {
    glds1(Ag + o0 + (16 + wid) * 1024, dst + (16 + wid) * 1024, voff);
    glds1(Ag + o1 + (16 + wid) * 1024, dst + 18 * 1024 + (16 + wid) * 1024, voff);
  }
}

__device__ __forceinline__ bool flags_ready(const u32* f, u32 target, int lane) {
  u32 v = __hip_atomic_load(f, __ATOMIC_RELAXED, __HIP_MEMORY_SCOPE_SYSTEM);
  return __all(v >= target);
}

__device__ void gemm_wg(const Params& p, u8* lds, u32 call, int g) {
  const int wid = __builtin_amdgcn_readfirstlane(threadIdx.x >> 6), lane = threadIdx.x & 63;
  const int xcd = g & 7, idx = g >> 3;
  const int tj = 2 * xcd + (idx & 1), ti = idx >> 1;
  const int row0 = (ti == NRB - 1) ? MM - TM : ti * TM;
  const int col0 = tj * TN;
  u8* ws = p.ws[p.rank];
  const u8* Ag = ws + OFF_STAGE + (size_t)(call & 1) * 2 * MAT_BYTES;
  const u8* Bg = Ag + MAT_BYTES;
  u32* tmo = (u32*)(ws + OFF_TMO);
  const u32 target = call * NRANK;
  const int fb = (lane & 3) == 0 ? 2 * tj : ((lane & 3) == 1 ? 2 * ti + 1 : (ti == NRB - 1 ? 2 * (NRB - 2) + 1 : 2 * ti + 1));
  const u32* fl = (const u32*)(ws + OFF_FLAG + (size_t)fb * 128);
  bool bad = false;

  auto spin = [&](int c) {
    const u32* f = fl + (size_t)c * 32 * 32;
    unsigned long long t0 = __builtin_amdgcn_s_memrealtime();
    while (!flags_ready(f, target, lane)) {
      if (__hip_atomic_load(tmo, __ATOMIC_RELAXED, __HIP_MEMORY_SCOPE_AGENT) == call ||
          __builtin_amdgcn_s_memrealtime() - t0 > DEADLINE_TICKS) {
        __hip_atomic_store(tmo, call, __ATOMIC_RELAXED, __HIP_MEMORY_SCOPE_AGENT);
        bad = true;
        break;
      }
      __builtin_amdgcn_s_sleep(4);
    }
    __builtin_amdgcn_fence(__ATOMIC_ACQUIRE, "");
  };

  const int wm = wid >> 2, wn = wid & 3;
  const int gq = lane >> 4, r = lane & 15;
  const int aoff = (gq >> 1) * 18 * 1024 + (wm * 9) * 1024 + (gq & 1) * 512 + r * 16;
  const int boff = 36 * 1024 + (gq >> 1) * 16 * 1024 + (wn * 4) * 1024 + (gq & 1) * 512 + r * 16;
  f32x4 acc[9][4];
#pragma unroll
  for (int i = 0; i < 9; ++i)
#pragma unroll
    for (int j = 0; j < 4; ++j) acc[i][j] = f32x4{0.f, 0.f, 0.f, 0.f};

  const u32 lds0 = (u32)(size_t)lds;
  const u32 voff = lane * 16;
  const u8* At = uniform_ptr(Ag + (size_t)(row0 >> 4) * 1024);
  const u8* Bt = uniform_ptr(Bg + (size_t)(col0 >> 4) * 1024);
  spin(0);
  gemm_stage(At, Bt, lds0, 0, wid, voff);
  for (int s = 0; s < NSTEP; ++s) {
    const bool more = s + 1 < NSTEP;
    const bool need = more && ((s + 1) & 3) == 0;
    bool ready = true;
    if (need) ready = flags_ready(fl + (size_t)((s + 1) >> 2) * 32 * 32, target, lane);
    WAIT_VM(0);
    __builtin_amdgcn_s_barrier();
    const u8* cur = lds + (s & 1) * STAGE_BYTES;
    const u32 nxt = lds0 + ((s + 1) & 1) * STAGE_BYTES;
    if (more && ready) {
      if (need) __builtin_amdgcn_fence(__ATOMIC_ACQUIRE, "");
      gemm_stage(At, Bt, nxt, s + 1, wid, voff);
    }
    i32x8 wf[4];
    i32x8 xf[3];
#define LDFRAG(dst, ptr) do { u32x4 lo_ = *(const u32x4*)(ptr); u32x4 hi_ = *(const u32x4*)((ptr) + 256); \
      dst = i32x8{(int)lo_[0], (int)lo_[1], (int)lo_[2], (int)lo_[3], (int)hi_[0], (int)hi_[1], (int)hi_[2], (int)hi_[3]}; } while (0)
#pragma unroll
    for (int j = 0; j < 4; ++j) LDFRAG(wf[j], cur + boff + j * 1024);
    LDFRAG(xf[0], cur + aoff);
    LDFRAG(xf[1], cur + aoff + 1024);
    __builtin_amdgcn_sched_barrier(0);
#pragma unroll
    for (int i = 0; i < 9; ++i) {
      if (i + 2 < 9) LDFRAG(xf[(i + 2) % 3], cur + aoff + (i + 2) * 1024);
#pragma unroll
      for (int j = 0; j < 4; ++j)
        acc[i][j] = __builtin_amdgcn_mfma_scale_f32_16x16x128_f8f6f4(wf[j], xf[i % 3], acc[i][j], 1, 1, 0, 0x7F7F7F7F, 0, 0x7F7F7F7F);
      __builtin_amdgcn_sched_barrier(0);
    }
#pragma unroll
    for (int i = 0; i < 9; ++i)
      asm volatile("" : "+v"(acc[i][0]), "+v"(acc[i][1]), "+v"(acc[i][2]), "+v"(acc[i][3]));
    if (more && !ready) {
      spin((s + 1) >> 2);
      gemm_stage(At, Bt, nxt, s + 1, wid, voff);
    }
  }
  const float sc = bad ? __builtin_nanf("") : p.sx[0] * p.sw[0];
  const float c2 = -sc * 1.4426950408889634f;
  const float lim = 60.0f * 1.4426950408889634f;
  const int rmin = ti * TM;
#pragma unroll
  for (int i = 0; i < 9; ++i) {
    const int row = row0 + wm * 144 + i * 16 + r;
    if (row >= rmin) {
      float* o = p.out + (size_t)row * NN + col0 + wn * 64 + gq * 4;
#pragma unroll
      for (int j = 0; j < 4; ++j) {
        f32x4 v;
#pragma unroll
        for (int e = 0; e < 4; ++e) {
          const float a = acc[i][j][e];
          const float y = a * sc;
          const float t = __builtin_amdgcn_fmed3f(a * c2, -lim, lim);
          const float ex = __builtin_amdgcn_exp2f(t);
          v[e] = y * __builtin_amdgcn_rcpf(1.0f + ex);
        }
        *(f32x4*)(o + j * 16) = v;
      }
    }
  }
}

__global__ void __launch_bounds__(512, 2) fused_kernel(Params p) {
  extern __shared__ __attribute__((aligned(16))) u8 lds[];
  u32* bc = (u32*)(lds + 2 * STAGE_BYTES);
  if (threadIdx.x == 0) {
    u32* cnt = (u32*)(p.ws[p.rank] + OFF_WGCNT + (size_t)blockIdx.x * 128);
    u32 v = __hip_atomic_load(cnt, __ATOMIC_RELAXED, __HIP_MEMORY_SCOPE_AGENT) + 1;
    __hip_atomic_store(cnt, v, __ATOMIC_RELAXED, __HIP_MEMORY_SCOPE_AGENT);
    *bc = v;
  }
  __syncthreads();
  const u32 call = __builtin_amdgcn_readfirstlane(*bc);
  if (blockIdx.x >= NPUSH) gemm_wg(p, lds, call, blockIdx.x - NPUSH);
  else if (blockIdx.x & 1) pusher_x(p, call, blockIdx.x >> 1);
  else pusher_w(p, lds, call, blockIdx.x >> 1);
}

extern "C" void kernel_launch(int rank, int nranks, ncclComm_t comm,
                              hipStream_t stream,
                              void* const* d_in, const int* in_sizes, int n_in,
                              void* d_out, int out_size,
                              const int* in_dtypes, int out_dtype,
                              void* const* d_out_all,
                              void* const* d_ws_all, size_t ws_bytes) {
  if (nranks != NRANK || (unsigned)rank >= (unsigned)NRANK || ws_bytes < WS_NEED || n_in < 4) {
    fprintf(stderr, "kernel_launch: unsupported configuration\n");
    return;
  }
  HIP_CHECK(hipFuncSetAttribute((const void*)fused_kernel, hipFuncAttributeMaxDynamicSharedMemorySize, LDS_BYTES));
  Params p;
  p.x = (const u8*)d_in[0]; p.w = (const u8*)d_in[1];
  p.sx = (const float*)d_in[2]; p.sw = (const float*)d_in[3];
  p.out = (float*)d_out;
  for (int i = 0; i < NRANK; ++i) p.ws[i] = (u8*)d_ws_all[i];
  p.rank = rank;
  fused_kernel<<<GRID, 512, LDS_BYTES, stream>>>(p);
  HIP_CHECK(hipGetLastError());
}
```

```cpp
#include <hip/hip_runtime.h>
#include <rccl/rccl.h>
#include <cstdio>

#define HIP_CHECK(expr) do { hipError_t _e = (expr); if (_e != hipSuccess) { \
    fprintf(stderr, "HIP error %s at %s:%d\n", hipGetErrorString(_e), __FILE__, __LINE__); \
    return; } } while (0)

typedef unsigned int u32;
typedef unsigned char u8;
typedef __attribute__((ext_vector_type(4))) u32 u32x4;
typedef __attribute__((ext_vector_type(2))) u32 u32x2;
typedef __attribute__((ext_vector_type(8))) int i32x8;
typedef __attribute__((ext_vector_type(4))) float f32x4;

constexpr int MM = 4096, NN = 4096, NRANK = 8, KSH = 512;
constexpr int CK = 32;
constexpr int NCH = KSH / CK;
constexpr int NSTEP = NCH * 2;
constexpr size_t SC_BYTES = 128u << 10;
constexpr int TM = 288, TN = 256, NRB = 15, NCB = 16;
constexpr int NPUSH = 16, GRID = NPUSH + NRB * NCB;
constexpr size_t OFF_WGCNT = 0;
constexpr size_t OFF_TMO = 32u << 10;
constexpr size_t OFF_FLAG = 64u << 10;
constexpr size_t OFF_STAGE = 1u << 20;
constexpr size_t MAT_BYTES = 16u << 20;
constexpr size_t WS_NEED = OFF_STAGE + 4 * MAT_BYTES;
constexpr unsigned long long DEADLINE_TICKS = 50000000ull;
constexpr int STAGE_BYTES = 68 * 1024;
constexpr int LDS_BYTES = 2 * STAGE_BYTES;

struct Params {
  const u8* x; const u8* w; const float* sx; const float* sw; float* out;
  u8* dst[NRANK];
  int rank;
};
__device__ __forceinline__ u8* lane_dst(const Params& p, int l) {
  u8* r = p.dst[0];
#pragma unroll
  for (int i = 1; i < NRANK; ++i) r = ((l & 7) == i) ? p.dst[i] : r;
  return r;
}

#define OUT_ST 2
#define STAMP(i) do {} while (0)
#define ACQ() __builtin_amdgcn_fence(__ATOMIC_ACQUIRE, "wavefront")
#define WAIT_VM(n) asm volatile("s_waitcnt vmcnt(" #n ")" ::: "memory")

__device__ __forceinline__ void st16_sys(u8* p, u32x4 v) {
  asm volatile("global_store_dwordx4 %0, %1, off sc0 sc1\n\ts_nop 1" :: "v"(p), "v"(v) : "memory");
}

__device__ __forceinline__ void tr4(u32 d0, u32 d1, u32 d2, u32 d3, u32& o0, u32& o1, u32& o2, u32& o3) {
  u32 t0 = __builtin_amdgcn_perm(d1, d0, 0x05010400u), t1 = __builtin_amdgcn_perm(d1, d0, 0x07030602u);
  u32 t2 = __builtin_amdgcn_perm(d3, d2, 0x05010400u), t3 = __builtin_amdgcn_perm(d3, d2, 0x07030602u);
  o0 = __builtin_amdgcn_perm(t2, t0, 0x05040100u); o1 = __builtin_amdgcn_perm(t2, t0, 0x07060302u);
  o2 = __builtin_amdgcn_perm(t3, t1, 0x05040100u); o3 = __builtin_amdgcn_perm(t3, t1, 0x07060302u);
}

#define LD16(dst, addr) asm volatile("global_load_dwordx4 %0, %1, off" : "=v"(dst) : "v"(addr) : "memory")
#define LD4(dst, addr) asm volatile("global_load_dword %0, %1, off" : "=v"(dst) : "v"(addr) : "memory")

#define PUSH_UNIT(c)                                                                                             \
    _Pragma("unroll") for (int i = 0; i < NRANK; ++i) {                                                          \
      u8* d = p.dst[i] + pbase + (size_t)(c) * SC_BYTES; st16_sys(d, c0); st16_sys(d + 1024, c1); }   \
    WAIT_VM(16);                                        \
    __builtin_amdgcn_s_barrier();                                                                                \
    if ((c) > 0 && threadIdx.x < NRANK)                                                                          \
      __hip_atomic_fetch_add(myflag + ((c) - 1) * 16 * 32, 1u, __ATOMIC_RELAXED, __HIP_MEMORY_SCOPE_SYSTEM);     \
    STAMP(1 + (c));
#define PUSH_END()                                                                                               \
  WAIT_VM(0);                                                                                                    \
  __builtin_amdgcn_s_barrier();                                                                                  \
  if (threadIdx.x < NRANK) __hip_atomic_fetch_add(myflag + (NCH - 1) * 16 * 32, 1u, __ATOMIC_RELAXED, __HIP_MEMORY_SCOPE_SYSTEM); \
  STAMP(40);

template <bool ISX>
__device__ __forceinline__ void pusher(const Params& p, u8* lds, u32* cnt, int q) {
  const int wid = __builtin_amdgcn_readfirstlane(threadIdx.x >> 6), lane = threadIdx.x & 63;
  u32* myflag = (u32*)(lane_dst(p, lane) + OFF_FLAG) + (2 * q + (ISX ? 1 : 0)) * 32;
  asm volatile("" : "+v"(myflag));
  u8* wreg = lds + wid * 8192;
  u32 cv;
  asm volatile("global_load_dword %0, %1, off sc1" : "=v"(cv) : "v"(cnt) : "memory");
  u32x4 r[32];
  const u8* src = ISX ? p.x + (size_t)(q * 512 + wid * 64 + (lane >> 3)) * KSH + (lane & 7) * 16
                      : p.w + (size_t)((lane >> 2) * 8) * NN + q * 512 + wid * 64 + (lane & 3) * 16;
#define LDSRC(j) LD16(r[j], ISX ? src + (size_t)((j) & 7) * 8 * KSH + ((j) >> 3) * 128 : src + (size_t)(((j) >> 3) * 128 + ((j) & 7)) * NN)
#pragma unroll
  for (int j = 0; j < 8; ++j) LDSRC(j);
  WAIT_VM(0);
  asm volatile("" : "+v"(cv));
  const u32 call = __builtin_amdgcn_readfirstlane(cv) + 1;
  STAMP(0);
  const size_t pbase = OFF_STAGE + (size_t)(call & 1) * 2 * MAT_BYTES + (ISX ? 0 : MAT_BYTES) + (size_t)p.rank * NCH * SC_BYTES + (size_t)(q * 32 + wid * 4) * 512 + lane * 16;
  u8* wp = ISX ? wreg + ((lane & 7) >> 1) * 2048 + (lane & 1) * 256 + (lane >> 3) * 16
               : wreg + (lane >> 4) * 2048 + (lane & 3) * 512 + ((lane >> 3) & 1) * 256 + ((lane >> 2) & 1) * 8;
#pragma unroll
  for (int b = 0; b < 4; ++b) {
    asm volatile("" : "+v"(r[8 * b]), "+v"(r[8 * b + 1]), "+v"(r[8 * b + 2]), "+v"(r[8 * b + 3]), "+v"(r[8 * b + 4]), "+v"(r[8 * b + 5]), "+v"(r[8 * b + 6]), "+v"(r[8 * b + 7]));
    if (ISX) {
#pragma unroll
      for (int i = 0; i < 8; ++i) *(u32x4*)(wp + (i >> 1) * 512 + (i & 1) * 128) = r[8 * b + i];
    } else {
#pragma unroll
      for (int dw = 0; dw < 4; ++dw) {
        u32 a0, a1, a2, a3, b0, b1, b2, b3;
        tr4(r[8 * b][dw], r[8 * b + 1][dw], r[8 * b + 2][dw], r[8 * b + 3][dw], a0, a1, a2, a3);
        tr4(r[8 * b + 4][dw], r[8 * b + 5][dw], r[8 * b + 6][dw], r[8 * b + 7][dw], b0, b1, b2, b3);
        *(u32x2*)(wp + dw * 64 + 0) = u32x2{a0, b0};
        *(u32x2*)(wp + dw * 64 + 16) = u32x2{a1, b1};
        *(u32x2*)(wp + dw * 64 + 32) = u32x2{a2, b2};
        *(u32x2*)(wp + dw * 64 + 48) = u32x2{a3, b3};
      }
    }
    asm volatile("s_waitcnt lgkmcnt(0)" ::: "memory");
#pragma unroll
    for (int cc = 0; cc < 4; ++cc) {
      u32x4 c0 = *(const u32x4*)(wreg + cc * 2048 + lane * 16);
      u32x4 c1 = *(const u32x4*)(wreg + cc * 2048 + 1024 + lane * 16);
      asm volatile("s_waitcnt lgkmcnt(0)" : "+v"(c0), "+v"(c1) :: "memory");
      {
        const int u = 4 * b + cc;
        const int j0 = u < 2 ? 8 + 3 * u : 14 + 2 * (u - 2), nj = u < 2 ? 3 : (u <= 10 ? 2 : 0);
#pragma unroll
        for (int j = j0; j < j0 + nj; ++j) LDSRC(j);
      }
      PUSH_UNIT(4 * b + cc);
    }
  }
  PUSH_END();
  if (threadIdx.x == 0) __hip_atomic_store(cnt, call, __ATOMIC_RELAXED, __HIP_MEMORY_SCOPE_AGENT);
}

template <typename T> __device__ __forceinline__ T* uniform_ptr(T* q) {
  unsigned long long v = (unsigned long long)q;
  u32 lo = __builtin_amdgcn_readfirstlane((u32)v), hi = __builtin_amdgcn_readfirstlane((u32)(v >> 32));
  return (T*)(((unsigned long long)hi << 32) | lo);
}
__device__ __forceinline__ void glds2(const u8* gbase, u32 ldsaddr, u32 voff) {
  u32 keep;
  asm volatile("s_mov_b32 %0, m0\n\ts_mov_b32 m0, %2\n\ts_nop 0\n\t"
               "global_load_lds_dwordx4 %1, %3 sc1\n\t"
               "global_load_lds_dwordx4 %1, %3 offset:1024 sc1\n\t"
               "s_mov_b32 m0, %0"
               : "=&s"(keep) : "v"(voff), "s"(ldsaddr), "s"(gbase) : "memory");
}
__device__ __forceinline__ void glds1(const u8* gbase, u32 ldsaddr, u32 voff) {
  u32 keep;
  asm volatile("s_mov_b32 %0, m0\n\ts_mov_b32 m0, %2\n\ts_nop 0\n\t"
               "global_load_lds_dwordx4 %1, %3 sc1\n\t"
               "s_mov_b32 m0, %0"
               : "=&s"(keep) : "v"(voff), "s"(ldsaddr), "s"(gbase) : "memory");
}
__device__ __forceinline__ void gemm_stage(const u8* Ag, const u8* Bg, u32 dst, int s, int wid, u32 voff) {
  const int c = s >> 1, s0 = (s & 1) * 4;
  const size_t o0 = (size_t)(s0 * NCH + c) * SC_BYTES + wid * 1024;
  constexpr size_t SS = (size_t)NCH * SC_BYTES;
#pragma unroll
  for (int g = 0; g < 4; ++g) {
    glds1(Ag + o0 + g * SS, dst + g * 9216 + wid * 1024, voff);
    glds1(Bg + o0 + g * SS, dst + 36864 + g * 8192 + wid * 1024, voff);
  }
  if (wid < 4) glds1(Ag + (size_t)((s0 + wid) * NCH + c) * SC_BYTES + 8192, dst + wid * 9216 + 8192, voff);
}

__device__ __forceinline__ bool flags_ready(const u32* f, u32 target, int lane) {
  u32 v = __hip_atomic_load(f, __ATOMIC_RELAXED, __HIP_MEMORY_SCOPE_SYSTEM);
  return __all(v >= target);
}

__device__ __forceinline__ void gemm_wg(const Params& p, u8* lds, u32* cnt, int g) {
  const int wid = __builtin_amdgcn_readfirstlane(threadIdx.x >> 6), lane = threadIdx.x & 63;
  const u32 call = __builtin_amdgcn_readfirstlane(__hip_atomic_load(cnt, __ATOMIC_RELAXED, __HIP_MEMORY_SCOPE_AGENT)) + 1;
  STAMP(0);
  const int xcd = g & 7, idx = g >> 3;
  const int tj = 2 * xcd + (idx & 1), ti = idx >> 1;
  const int row0 = (ti == NRB - 1) ? MM - TM : ti * TM;
  const int col0 = tj * TN;
  u8* ws = p.dst[7];
  const u8* Ag = ws + OFF_STAGE + (size_t)(call & 1) * 2 * MAT_BYTES;
  const u8* Bg = Ag + MAT_BYTES;
  u32* tmo = (u32*)(ws + OFF_TMO);
  const u32 target = call * NRANK;
  const int fb = (lane & 3) == 0 ? 2 * (col0 >> 9) : ((lane & 3) == 1 ? 2 * (row0 >> 9) + 1 : 2 * ((row0 + TM - 1) >> 9) + 1);
  const u32* fl = (const u32*)(ws + OFF_FLAG + (size_t)fb * 128);
  bool bad = false;

  auto spin = [&](int c) {
    const u32* f = fl + (size_t)c * 16 * 32;
    unsigned long long t0 = __builtin_amdgcn_s_memrealtime();
    for (u32 it = 1; !flags_ready(f, target, lane); ++it) {
      if ((it & 15) == 0) {
        if (__hip_atomic_load(tmo, __ATOMIC_RELAXED, __HIP_MEMORY_SCOPE_AGENT) == call ||
            __builtin_amdgcn_s_memrealtime() - t0 > DEADLINE_TICKS) {
          __hip_atomic_store(tmo, call, __ATOMIC_RELAXED, __HIP_MEMORY_SCOPE_AGENT);
          bad = true;
          break;
        }
      }
      __builtin_amdgcn_s_sleep(2);
    }
    ACQ();
  };

  const int wm = wid >> 2, wn = wid & 3;
  const int gq = lane >> 4, r = lane & 15;
  const int aoff = gq * 9216 + (wm * 9) * 512 + r * 16;
  const int boff = 36864 + gq * 8192 + (wn * 4) * 512 + r * 16;
  f32x4 acc[9][4];
#pragma unroll
  for (int i = 0; i < 9; ++i)
#pragma unroll
    for (int j = 0; j < 4; ++j) acc[i][j] = f32x4{0.f, 0.f, 0.f, 0.f};

  const u32 lds0 = (u32)(size_t)lds;
  const u32 voff = lane * 16;
  const u8* At = uniform_ptr(Ag + (size_t)(row0 >> 4) * 512);
  const u8* Bt = uniform_ptr(Bg + (size_t)(col0 >> 4) * 512);
  spin(0);
  gemm_stage(At, Bt, lds0, 0, wid, voff);
  for (int s = 0; s < NSTEP; ++s) {
    const bool more = s + 1 < NSTEP;
    const bool need = more && ((s + 1) & 1) == 0;
    bool ready = true;
    if (need) ready = flags_ready(fl + (size_t)((s + 1) >> 1) * 16 * 32, target, lane);
    WAIT_VM(0);
    __builtin_amdgcn_s_barrier();
    STAMP(1 + s);
    const u8* cur = lds + (s & 1) * STAGE_BYTES;
    const u32 nxt = lds0 + ((s + 1) & 1) * STAGE_BYTES;
    if (more && ready) {
      if (need) ACQ();
      gemm_stage(At, Bt, nxt, s + 1, wid, voff);
    }
    i32x8 wf[4];
    i32x8 xf[3];
#define LDFRAG(dst, ptr) do { u32x4 lo_ = *(const u32x4*)(ptr); u32x4 hi_ = *(const u32x4*)((ptr) + 256); \
      dst = i32x8{(int)lo_[0], (int)lo_[1], (int)lo_[2], (int)lo_[3], (int)hi_[0], (int)hi_[1], (int)hi_[2], (int)hi_[3]}; } while (0)
#pragma unroll
    for (int j = 0; j < 4; ++j) LDFRAG(wf[j], cur + boff + j * 512);
    LDFRAG(xf[0], cur + aoff);
    LDFRAG(xf[1], cur + aoff + 512);
    __builtin_amdgcn_sched_barrier(0);
#pragma unroll
    for (int i = 0; i < 9; ++i) {
      if (i + 2 < 9) LDFRAG(xf[(i + 2) % 3], cur + aoff + (i + 2) * 512);
#pragma unroll
      for (int j = 0; j < 4; ++j)
        acc[i][j] = __builtin_amdgcn_mfma_scale_f32_16x16x128_f8f6f4(wf[j], xf[i % 3], acc[i][j], 1, 1, 0, 0x7F7F7F7F, 0, 0x7F7F7F7F);
      __builtin_amdgcn_sched_barrier(0);
    }
#pragma unroll
    for (int i = 0; i < 9; ++i)
      asm volatile("" : "+v"(acc[i][0]), "+v"(acc[i][1]), "+v"(acc[i][2]), "+v"(acc[i][3]));
    if (more && !ready) {
      spin((s + 1) >> 1);
      gemm_stage(At, Bt, nxt, s + 1, wid, voff);
    }
  }
  STAMP(40);
  const float sc = bad ? __builtin_nanf("") : p.sx[0] * p.sw[0];
  const float c2 = -sc * 1.4426950408889634f;
  const float lim = 60.0f * 1.4426950408889634f;
  const int rmin = ti * TM;
#pragma unroll
  for (int i = 0; i < 9; ++i) {
    const int row = row0 + wm * 144 + i * 16 + r;
    if (row >= rmin) {
      float* o = p.out + (size_t)row * NN + col0 + wn * 64 + gq * 4;
#pragma unroll
      for (int j = 0; j < 4; ++j) {
        f32x4 v;
#pragma unroll
        for (int e = 0; e < 4; ++e) {
          const float a = acc[i][j][e];
          const float y = a * sc;
          const float t = __builtin_amdgcn_fmed3f(a * c2, -lim, lim);
          const float ex = __builtin_amdgcn_exp2f(t);
          v[e] = y * __builtin_amdgcn_rcpf(1.0f + ex);
        }
        asm volatile("global_store_dwordx4 %0, %1, off sc0 sc1\n\ts_nop 1" :: "v"(o + j * 16), "v"(v) : "memory");
      }
    }
  }
  STAMP(41);
  if (threadIdx.x == 0) __hip_atomic_store(cnt, call, __ATOMIC_RELAXED, __HIP_MEMORY_SCOPE_AGENT);
}

__global__ void __launch_bounds__(512, 2) fused_kernel(Params p) {
  extern __shared__ __attribute__((aligned(16))) u8 lds[];
  u32* cnt = (u32*)(p.dst[7] + OFF_WGCNT + (size_t)blockIdx.x * 128);
  if (blockIdx.x >= NPUSH) gemm_wg(p, lds, cnt, blockIdx.x - NPUSH);
  else if (blockIdx.x & 1) pusher<true>(p, lds, cnt, blockIdx.x >> 1);
  else pusher<false>(p, lds, cnt, blockIdx.x >> 1);
}

extern "C" void kernel_launch(int rank, int nranks, ncclComm_t comm,
                              hipStream_t stream,
                              void* const* d_in, const int* in_sizes, int n_in,
                              void* d_out, int out_size,
                              const int* in_dtypes, int out_dtype,
                              void* const* d_out_all,
                              void* const* d_ws_all, size_t ws_bytes) {
  if (nranks != NRANK || (unsigned)rank >= (unsigned)NRANK || ws_bytes < WS_NEED || n_in < 4) {
    fprintf(stderr, "kernel_launch: unsupported configuration\n");
    return;
  }
  HIP_CHECK(hipFuncSetAttribute((const void*)fused_kernel, hipFuncAttributeMaxDynamicSharedMemorySize, LDS_BYTES));
  Params p;
  p.x = (const u8*)d_in[0]; p.w = (const u8*)d_in[1];
  p.sx = (const float*)d_in[2]; p.sw = (const float*)d_in[3];
  p.out = (float*)d_out;
  for (int i = 0; i < NRANK; ++i) p.dst[i] = (u8*)d_ws_all[(rank + 1 + i) & 7];
  p.rank = rank;
  fused_kernel<<<GRID, 512, LDS_BYTES, stream>>>(p);
  HIP_CHECK(hipGetLastError());
}
```

```cpp
#include <hip/hip_runtime.h>
#include <rccl/rccl.h>
#include <cstdio>

#define HIP_CHECK(expr) do { hipError_t _e = (expr); if (_e != hipSuccess) { \
    fprintf(stderr, "HIP error %s at %s:%d\n", hipGetErrorString(_e), __FILE__, __LINE__); \
    return; } } while (0)

typedef unsigned int u32;
typedef unsigned char u8;
typedef __attribute__((address_space(1))) unsigned int gu32;
typedef __attribute__((ext_vector_type(4))) u32 u32x4;
typedef __attribute__((ext_vector_type(2))) u32 u32x2;
typedef __attribute__((ext_vector_type(8))) int i32x8;
typedef __attribute__((ext_vector_type(4))) float f32x4;

constexpr int MM = 4096, NN = 4096, NRANK = 8, KSH = 512;
constexpr int CK = 32;
constexpr int NCH = KSH / CK;
constexpr int NSTEP = NCH * 2;
constexpr size_t SC_BYTES = 128u << 10;
constexpr int TM = 320, TN = 256, NRB = 13, NCB = 16, MF = TM / 32;
constexpr int NPUSH = 20, GRID = NPUSH + NRB * NCB;
constexpr size_t OFF_WGCNT = 0;
constexpr size_t OFF_TMO = 32u << 10;
constexpr size_t OFF_TICKET = (32u << 10) + 4096;
constexpr size_t OFF_FLAG = 64u << 10;
constexpr size_t OFF_STAGE = 1u << 20;
constexpr size_t MAT_BYTES = 16u << 20;
constexpr size_t WS_NEED = OFF_STAGE + 4 * MAT_BYTES;
constexpr unsigned long long DEADLINE_TICKS = 50000000ull;
constexpr int ASRC = TM * 32;
constexpr int STAGE_BYTES = 4 * ASRC + 32768;
constexpr int LDS_BYTES = 2 * STAGE_BYTES;

struct Params {
  const u8* x; const u8* w; const float* sx; const float* sw; float* out;
  u8* dst[NRANK];
  int rank;
};
__device__ __forceinline__ u8* lane_dst(const Params& p, int l) {
  u8* r = p.dst[0];
#pragma unroll
  for (int i = 1; i < NRANK; ++i) r = ((l & 7) == i) ? p.dst[i] : r;
  return r;
}

#define OUT_ST 2
#define STAMP(i) do {} while (0)
#define ACQ() __builtin_amdgcn_fence(__ATOMIC_ACQUIRE, "wavefront")
#define WAIT_VM(n) asm volatile("s_waitcnt vmcnt(" #n ")" ::: "memory")

__device__ __forceinline__ void st16_sys(u8* p, u32x4 v) {
  asm volatile("global_store_dwordx4 %0, %1, off sc0 sc1\n\ts_nop 1" :: "v"(p), "v"(v) : "memory");
}

__device__ __forceinline__ void tr4(u32 d0, u32 d1, u32 d2, u32 d3, u32& o0, u32& o1, u32& o2, u32& o3) {
  u32 t0 = __builtin_amdgcn_perm(d1, d0, 0x05010400u), t1 = __builtin_amdgcn_perm(d1, d0, 0x07030602u);
  u32 t2 = __builtin_amdgcn_perm(d3, d2, 0x05010400u), t3 = __builtin_amdgcn_perm(d3, d2, 0x07030602u);
  o0 = __builtin_amdgcn_perm(t2, t0, 0x05040100u); o1 = __builtin_amdgcn_perm(t2, t0, 0x07060302u);
  o2 = __builtin_amdgcn_perm(t3, t1, 0x05040100u); o3 = __builtin_amdgcn_perm(t3, t1, 0x07060302u);
}

#define LD16(dst, addr) asm volatile("global_load_dwordx4 %0, %1, off" : "=v"(dst) : "v"(addr) : "memory")
#define LD4(dst, addr) asm volatile("global_load_dword %0, %1, off" : "=v"(dst) : "v"(addr) : "memory")

constexpr int NUNIT = NCH * 32;
template <bool ISX>
__device__ __forceinline__ void pusher(const Params& p, u8* lds, u32* cnt) {
  const int wid = __builtin_amdgcn_readfirstlane(threadIdx.x >> 6), lane = threadIdx.x & 63, w4 = wid & 3;
  gu32* myflag = (gu32*)(lane_dst(p, lane) + OFF_FLAG);
  asm volatile("" : "+v"(myflag));
  u8* wreg = lds + wid * 1024;
  volatile __attribute__((address_space(3))) u32* tslot = (volatile __attribute__((address_space(3))) u32*)(lds + 8192);
  u32 cv, tk = 0;
  const u32 one = 1;
  asm volatile("global_load_dword %0, %1, off sc1" : "=v"(cv) : "v"(cnt) : "memory");
  u32x4 n0; u32 d0, d1, d2, d3;
  const u8* src = ISX ? p.x + (size_t)(w4 * 32 + (lane >> 5) * 16 + (lane & 15)) * KSH + ((lane >> 4) & 1) * 16
                      : p.w + (size_t)((lane >> 3) * 4) * NN + w4 * 32 + (lane & 7) * 4;
  u8* wp = wreg + ((lane & 7) >> 2) * 512 + (lane >> 5) * 256 + (lane & 3) * 64 + ((lane >> 3) & 3) * 4;
  auto issue = [&](int t) {
    const int c = t >> 5, b = t & 31;
    if (ISX) { LD16(n0, src + (size_t)b * 128 * KSH + c * CK); }
    else { const u8* a = src + (size_t)c * CK * NN + b * 128; LD4(d0, a); LD4(d1, a + NN); LD4(d2, a + 2 * NN); LD4(d3, a + 3 * NN); }
  };
#define PIN() do { if (ISX) asm volatile("" : "+v"(n0), "+v"(tk)); else asm volatile("" : "+v"(d0), "+v"(d1), "+v"(d2), "+v"(d3)); } while (0)
  int tcur = blockIdx.x, tnext = blockIdx.x + NPUSH;
  issue(tcur);
  WAIT_VM(0);
  PIN();
  asm volatile("" : "+v"(cv));
  const u32 call = __builtin_amdgcn_readfirstlane(cv) + 1;
  STAMP(0);
  u32* tick = (u32*)(p.dst[7] + OFF_TICKET + (call & 3) * 128);
  if (blockIdx.x == 0 && threadIdx.x == 64)
    __hip_atomic_store((u32*)(p.dst[7] + OFF_TICKET + ((call + 2) & 3) * 128), 0u, __ATOMIC_RELAXED, __HIP_MEMORY_SCOPE_AGENT);
  const size_t pbase = OFF_STAGE + (size_t)(call & 1) * 2 * MAT_BYTES + (ISX ? 0 : MAT_BYTES) + (size_t)p.rank * NCH * SC_BYTES + (size_t)(w4 * 2) * 512 + lane * 16;
  int prev = -1;
  for (int it = 0;; ++it) {
    u32x4 c0;
    if (ISX) {
      c0 = n0;
      asm volatile("" : "+v"(c0));
    } else {
      u32 a0, a1, a2, a3;
      tr4(d0, d1, d2, d3, a0, a1, a2, a3);
      *(u32*)(wp + 0) = a0; *(u32*)(wp + 16) = a1; *(u32*)(wp + 32) = a2; *(u32*)(wp + 48) = a3;
      asm volatile("s_waitcnt lgkmcnt(0)" ::: "memory");
      c0 = *(const u32x4*)(wreg + lane * 16);
      asm volatile("s_waitcnt lgkmcnt(0)" : "+v"(c0) :: "memory");
    }
    const bool hasnext = tnext < NUNIT;
    issue(hasnext ? tnext : tcur);
    if (ISX && hasnext && threadIdx.x == 0)
      asm volatile("global_atomic_add %0, %1, %2, off sc0 sc1" : "=v"(tk) : "v"(tick), "v"(one) : "memory");
    const size_t doff = pbase + (size_t)(tcur >> 5) * SC_BYTES + (size_t)(tcur & 31) * 4096;
#pragma unroll
    for (int i = 0; i < NRANK; ++i) st16_sys(p.dst[i] + doff, c0);
    WAIT_VM(8);
    PIN();
    if (ISX && hasnext && threadIdx.x == 0) tslot[it & 1] = tk + 2 * NPUSH;
    asm volatile("s_waitcnt lgkmcnt(0)" ::: "memory");
    __builtin_amdgcn_s_barrier();
    if (ISX && prev >= 0 && threadIdx.x < NRANK) __hip_atomic_fetch_add(myflag + prev * 32, 1u, __ATOMIC_RELAXED, __HIP_MEMORY_SCOPE_SYSTEM);
    prev = tcur;
    STAMP(1 + (it < 38 ? it : 38));
    if (!hasnext) break;
    tcur = tnext;
    tnext = __builtin_amdgcn_readfirstlane(tslot[it & 1]);
  }
  WAIT_VM(0);
  __builtin_amdgcn_s_barrier();
  if (ISX && threadIdx.x < NRANK) __hip_atomic_fetch_add(myflag + prev * 32, 1u, __ATOMIC_RELAXED, __HIP_MEMORY_SCOPE_SYSTEM);
  STAMP(40);
  if (threadIdx.x == 0) __hip_atomic_store(cnt, call, __ATOMIC_RELAXED, __HIP_MEMORY_SCOPE_AGENT);
}

template <typename T> __device__ __forceinline__ T* uniform_ptr(T* q) {
  unsigned long long v = (unsigned long long)q;
  u32 lo = __builtin_amdgcn_readfirstlane((u32)v), hi = __builtin_amdgcn_readfirstlane((u32)(v >> 32));
  return (T*)(((unsigned long long)hi << 32) | lo);
}
__device__ __forceinline__ void glds2(const u8* gbase, u32 ldsaddr, u32 voff) {
  u32 keep;
  asm volatile("s_mov_b32 %0, m0\n\ts_mov_b32 m0, %2\n\ts_nop 0\n\t"
               "global_load_lds_dwordx4 %1, %3 sc1\n\t"
               "global_load_lds_dwordx4 %1, %3 offset:1024 sc1\n\t"
               "s_mov_b32 m0, %0"
               : "=&s"(keep) : "v"(voff), "s"(ldsaddr), "s"(gbase) : "memory");
}
__device__ __forceinline__ void glds1(const u8* gbase, u32 ldsaddr, u32 voff) {
  u32 keep;
  asm volatile("s_mov_b32 %0, m0\n\ts_mov_b32 m0, %2\n\ts_nop 0\n\t"
               "global_load_lds_dwordx4 %1, %3 sc1\n\t"
               "s_mov_b32 m0, %0"
               : "=&s"(keep) : "v"(voff), "s"(ldsaddr), "s"(gbase) : "memory");
}
__device__ __forceinline__ void gemm_stage(const u8* Ag, const u8* Bg, u32 dst, int s, int wid, u32 voff) {
  const int c = s >> 1, s0 = (s & 1) * 4;
  const size_t o0 = (size_t)(s0 * NCH + c) * SC_BYTES + wid * 1024;
  constexpr size_t SS = (size_t)NCH * SC_BYTES;
#pragma unroll
  for (int g = 0; g < 4; ++g) {
    glds1(Ag + o0 + g * SS, dst + g * ASRC + wid * 1024, voff);
    glds1(Bg + o0 + g * SS, dst + 4 * ASRC + g * 8192 + wid * 1024, voff);
  }
  glds1(Ag + (size_t)((s0 + (wid >> 1)) * NCH + c) * SC_BYTES + (8 + (wid & 1)) * 1024, dst + (wid >> 1) * ASRC + (8 + (wid & 1)) * 1024, voff);
}

__device__ __forceinline__ bool flags_ready(const u32* f, u32 target, int lane) {
  u32 v = __hip_atomic_load(f, __ATOMIC_RELAXED, __HIP_MEMORY_SCOPE_SYSTEM);
  return __all(v >= target);
}

__device__ __forceinline__ void gemm_wg(const Params& p, u8* lds, u32* cnt, int g) {
  const int wid = __builtin_amdgcn_readfirstlane(threadIdx.x >> 6), lane = threadIdx.x & 63;
  const u32 call = __builtin_amdgcn_readfirstlane(__hip_atomic_load(cnt, __ATOMIC_RELAXED, __HIP_MEMORY_SCOPE_AGENT)) + 1;
  STAMP(0);
  const int xcd = g & 7, idx = g >> 3;
  const int tl = xcd * (NRB * 2) + idx;
  const int tj = (tl / (NRB * 4)) * 4 + (tl & 3), ti = (tl % (NRB * 4)) >> 2;
  const int row0 = (ti == NRB - 1) ? MM - TM : ti * TM;
  const int col0 = tj * TN;
  u8* ws = p.dst[7];
  const u8* Ag = ws + OFF_STAGE + (size_t)(call & 1) * 2 * MAT_BYTES;
  const u8* Bg = Ag + MAT_BYTES;
  u32* tmo = (u32*)(ws + OFF_TMO);
  const u32 target = call * NRANK;
  const int l7 = lane & 7;
  const int fb = l7 < 2 ? 2 * tj + l7 : (row0 >> 7) + (l7 > 4 ? 2 : l7 - 2);
  const u32* fl = (const u32*)(ws + OFF_FLAG + (size_t)fb * 128);
  bool bad = false;

  auto spin = [&](int c) {
    const u32* f = fl + (size_t)c * 32 * 32;
    unsigned long long t0 = __builtin_amdgcn_s_memrealtime();
    for (u32 it = 1; !flags_ready(f, target, lane); ++it) {
      if ((it & 15) == 0) {
        if (__hip_atomic_load(tmo, __ATOMIC_RELAXED, __HIP_MEMORY_SCOPE_AGENT) == call ||
            __builtin_amdgcn_s_memrealtime() - t0 > DEADLINE_TICKS) {
          __hip_atomic_store(tmo, call, __ATOMIC_RELAXED, __HIP_MEMORY_SCOPE_AGENT);
          bad = true;
          break;
        }
      }
      __builtin_amdgcn_s_sleep(2);
    }
    ACQ();
  };

  const int wm = wid >> 2, wn = wid & 3;
  const int gq = lane >> 4, r = lane & 15;
  const int aoff = gq * ASRC + (wm * MF) * 512 + r * 16;
  const int boff = 4 * ASRC + gq * 8192 + (wn * 4) * 512 + r * 16;
  f32x4 acc[MF][4];
#pragma unroll
  for (int i = 0; i < MF; ++i)
#pragma unroll
    for (int j = 0; j < 4; ++j) acc[i][j] = f32x4{0.f, 0.f, 0.f, 0.f};

  const u32 lds0 = (u32)(size_t)lds;
  const u32 voff = lane * 16;
  const u8* At = uniform_ptr(Ag + (size_t)(row0 >> 4) * 512);
  const u8* Bt = uniform_ptr(Bg + (size_t)(col0 >> 4) * 512);
  spin(0);
  gemm_stage(At, Bt, lds0, 0, wid, voff);
  for (int s = 0; s < NSTEP; ++s) {
    const bool more = s + 1 < NSTEP;
    const bool need = more && ((s + 1) & 1) == 0;
    bool ready = true;
    if (need) ready = flags_ready(fl + (size_t)((s + 1) >> 1) * 32 * 32, target, lane);
    WAIT_VM(0);
    __builtin_amdgcn_s_barrier();
    STAMP(1 + s);
    const u8* cur = lds + (s & 1) * STAGE_BYTES;
    const u32 nxt = lds0 + ((s + 1) & 1) * STAGE_BYTES;
    if (more && ready) {
      if (need) ACQ();
      gemm_stage(At, Bt, nxt, s + 1, wid, voff);
    }
    i32x8 wf[4];
    i32x8 xf[3];
#define LDFRAG(dst, ptr) do { u32x4 lo_ = *(const u32x4*)(ptr); u32x4 hi_ = *(const u32x4*)((ptr) + 256); \
      dst = i32x8{(int)lo_[0], (int)lo_[1], (int)lo_[2], (int)lo_[3], (int)hi_[0], (int)hi_[1], (int)hi_[2], (int)hi_[3]}; } while (0)
#pragma unroll
    for (int j = 0; j < 4; ++j) LDFRAG(wf[j], cur + boff + j * 512);
    LDFRAG(xf[0], cur + aoff);
    LDFRAG(xf[1], cur + aoff + 512);
    __builtin_amdgcn_sched_barrier(0);
#pragma unroll
    for (int i = 0; i < MF; ++i) {
      if (i + 2 < MF) LDFRAG(xf[(i + 2) % 3], cur + aoff + (i + 2) * 512);
#pragma unroll
      for (int j = 0; j < 4; ++j)
        acc[i][j] = __builtin_amdgcn_mfma_scale_f32_16x16x128_f8f6f4(wf[j], xf[i % 3], acc[i][j], 1, 1, 0, 0x7F7F7F7F, 0, 0x7F7F7F7F);
      __builtin_amdgcn_sched_barrier(0);
    }
#pragma unroll
    for (int i = 0; i < MF; ++i)
      asm volatile("" : "+v"(acc[i][0]), "+v"(acc[i][1]), "+v"(acc[i][2]), "+v"(acc[i][3]));
    if (more && !ready) {
      spin((s + 1) >> 1);
      gemm_stage(At, Bt, nxt, s + 1, wid, voff);
    }
  }
  STAMP(40);
  const float sc = bad ? __builtin_nanf("") : p.sx[0] * p.sw[0];
  const float c2 = -sc * 1.4426950408889634f;
  const float lim = 60.0f * 1.4426950408889634f;
  const int rmin = ti * TM;
#pragma unroll
  for (int i = 0; i < MF; ++i) {
    const int row = row0 + wm * (TM / 2) + i * 16 + r;
    if (row >= rmin) {
      float* o = p.out + (size_t)row * NN + col0 + wn * 64 + gq * 4;
#pragma unroll
      for (int j = 0; j < 4; ++j) {
        f32x4 v;
#pragma unroll
        for (int e = 0; e < 4; ++e) {
          const float a = acc[i][j][e];
          const float y = a * sc;
          const float t = __builtin_amdgcn_fmed3f(a * c2, -lim, lim);
          const float ex = __builtin_amdgcn_exp2f(t);
          v[e] = y * __builtin_amdgcn_rcpf(1.0f + ex);
        }
        asm volatile("global_store_dwordx4 %0, %1, off sc0 sc1\n\ts_nop 1" :: "v"(o + j * 16), "v"(v) : "memory");
      }
    }
  }
  STAMP(41);
  if (threadIdx.x == 0) __hip_atomic_store(cnt, call, __ATOMIC_RELAXED, __HIP_MEMORY_SCOPE_AGENT);
}

__global__ void __launch_bounds__(512, 2) fused_kernel(Params p) {
  extern __shared__ __attribute__((aligned(16))) u8 lds[];
  u32* cnt = (u32*)(p.dst[7] + OFF_WGCNT + (size_t)blockIdx.x * 128);
  if (blockIdx.x >= NPUSH) gemm_wg(p, lds, cnt, blockIdx.x - NPUSH);
  else if (__builtin_amdgcn_readfirstlane(threadIdx.x) < 256) pusher<true>(p, lds, cnt);
  else pusher<false>(p, lds, cnt);
}

extern "C" void kernel_launch(int rank, int nranks, ncclComm_t comm,
                              hipStream_t stream,
                              void* const* d_in, const int* in_sizes, int n_in,
                              void* d_out, int out_size,
                              const int* in_dtypes, int out_dtype,
                              void* const* d_out_all,
                              void* const* d_ws_all, size_t ws_bytes) {
  if (nranks != NRANK || (unsigned)rank >= (unsigned)NRANK || ws_bytes < WS_NEED || n_in < 4) {
    fprintf(stderr, "kernel_launch: unsupported configuration\n");
    return;
  }
  HIP_CHECK(hipFuncSetAttribute((const void*)fused_kernel, hipFuncAttributeMaxDynamicSharedMemorySize, LDS_BYTES));
  Params p;
  p.x = (const u8*)d_in[0]; p.w = (const u8*)d_in[1];
  p.sx = (const float*)d_in[2]; p.sw = (const float*)d_in[3];
  p.out = (float*)d_out;
  for (int i = 0; i < NRANK; ++i) p.dst[i] = (u8*)d_ws_all[(rank + 1 + i) & 7];
  p.rank = rank;
  fused_kernel<<<GRID, 512, LDS_BYTES, stream>>>(p);
  HIP_CHECK(hipGetLastError());
}
```

```cpp
#include <hip/hip_runtime.h>
#include <rccl/rccl.h>
#include <cstdio>

#define HIP_CHECK(expr) do { hipError_t _e = (expr); if (_e != hipSuccess) { \
    fprintf(stderr, "HIP error %s at %s:%d\n", hipGetErrorString(_e), __FILE__, __LINE__); \
    return; } } while (0)

typedef unsigned int u32;
typedef unsigned char u8;
typedef __attribute__((address_space(1))) unsigned int gu32;
typedef __attribute__((ext_vector_type(4))) u32 u32x4;
typedef __attribute__((ext_vector_type(2))) u32 u32x2;
typedef __attribute__((ext_vector_type(8))) int i32x8;
typedef __attribute__((ext_vector_type(4))) float f32x4;

constexpr int MM = 4096, NN = 4096, NRANK = 8, KSH = 512;
constexpr int CK = 32;
constexpr int NCH = KSH / CK;
constexpr int NSTEP = NCH * 2;
constexpr size_t SC_BYTES = 128u << 10;
constexpr int TM = 320, TN = 256, NRB = 13, NCB = 16, MF = TM / 32;
constexpr int NPUSH = 22, GRID = NPUSH + NRB * NCB;
constexpr size_t OFF_WGCNT = 0;
constexpr size_t OFF_TMO = 32u << 10;
constexpr size_t OFF_TICKET = (32u << 10) + 4096;
constexpr size_t OFF_FLAG = 64u << 10;
constexpr size_t OFF_STAGE = 1u << 20;
constexpr size_t MAT_BYTES = 16u << 20;
constexpr size_t WS_NEED = OFF_STAGE + 4 * MAT_BYTES;
constexpr unsigned long long DEADLINE_TICKS = 50000000ull;
constexpr int ASRC = TM * 32;
constexpr int STAGE_BYTES = 4 * ASRC + 32768;
constexpr int LDS_BYTES = 2 * STAGE_BYTES;

struct Params {
  const u8* x; const u8* w; const float* sx; const float* sw; float* out;
  u8* dst[NRANK];
  int rank;
};
__device__ __forceinline__ u8* lane_dst(const Params& p, int l) {
  u8* r = p.dst[0];
#pragma unroll
  for (int i = 1; i < NRANK; ++i) r = ((l & 7) == i) ? p.dst[i] : r;
  return r;
}

#define STAMP(i) do {} while (0)
#define ACQ() __builtin_amdgcn_fence(__ATOMIC_ACQUIRE, "wavefront")
#define WAIT_VM(n) asm volatile("s_waitcnt vmcnt(" #n ")" ::: "memory")

__device__ __forceinline__ void st16_sys(u8* p, u32x4 v) {
  asm volatile("global_store_dwordx4 %0, %1, off sc0 sc1\n\ts_nop 1" :: "v"(p), "v"(v) : "memory");
}

__device__ __forceinline__ void tr4(u32 d0, u32 d1, u32 d2, u32 d3, u32& o0, u32& o1, u32& o2, u32& o3) {
  u32 t0 = __builtin_amdgcn_perm(d1, d0, 0x05010400u), t1 = __builtin_amdgcn_perm(d1, d0, 0x07030602u);
  u32 t2 = __builtin_amdgcn_perm(d3, d2, 0x05010400u), t3 = __builtin_amdgcn_perm(d3, d2, 0x07030602u);
  o0 = __builtin_amdgcn_perm(t2, t0, 0x05040100u); o1 = __builtin_amdgcn_perm(t2, t0, 0x07060302u);
  o2 = __builtin_amdgcn_perm(t3, t1, 0x05040100u); o3 = __builtin_amdgcn_perm(t3, t1, 0x07060302u);
}

#define LD16(dst, addr) asm volatile("global_load_dwordx4 %0, %1, off" : "=v"(dst) : "v"(addr) : "memory")
#define LD4(dst, addr) asm volatile("global_load_dword %0, %1, off" : "=v"(dst) : "v"(addr) : "memory")

constexpr int NUNIT = NCH * 32;
template <bool ISX>
__device__ __forceinline__ void pusher(const Params& p, u8* lds, u32* cnt) {
  const int wid = __builtin_amdgcn_readfirstlane(threadIdx.x >> 6), lane = threadIdx.x & 63, w4 = wid & 3;
  gu32* myflag = (gu32*)(lane_dst(p, lane) + OFF_FLAG);
  asm volatile("" : "+v"(myflag));
  u8* wreg = lds + wid * 1024;
  volatile __attribute__((address_space(3))) u32* tslot = (volatile __attribute__((address_space(3))) u32*)(lds + 8192);
  u32 cv, tk = 0;
  const u32 one = 1;
  asm volatile("global_load_dword %0, %1, off sc1" : "=v"(cv) : "v"(cnt) : "memory");
  u32x4 n0; u32 d0, d1, d2, d3;
  const u8* src = ISX ? p.x + (size_t)(w4 * 32 + (lane >> 5) * 16 + (lane & 15)) * KSH + ((lane >> 4) & 1) * 16
                      : p.w + (size_t)((lane >> 3) * 4) * NN + w4 * 32 + (lane & 7) * 4;
  u8* wp = wreg + ((lane & 7) >> 2) * 512 + (lane >> 5) * 256 + (lane & 3) * 64 + ((lane >> 3) & 3) * 4;
  auto issue = [&](int t) {
    const int c = t >> 5, b = t & 31;
    if (ISX) { LD16(n0, src + (size_t)b * 128 * KSH + c * CK); }
    else { const u8* a = src + (size_t)c * CK * NN + b * 128; LD4(d0, a); LD4(d1, a + NN); LD4(d2, a + 2 * NN); LD4(d3, a + 3 * NN); }
  };
#define PIN() do { if (ISX) asm volatile("" : "+v"(n0), "+v"(tk)); else asm volatile("" : "+v"(d0), "+v"(d1), "+v"(d2), "+v"(d3)); } while (0)
  int tcur = blockIdx.x, tnext = blockIdx.x + NPUSH;
  issue(tcur);
  WAIT_VM(0);
  PIN();
  asm volatile("" : "+v"(cv));
  const u32 call = __builtin_amdgcn_readfirstlane(cv) + 1;
  STAMP(0);
  u32* tick = (u32*)(p.dst[7] + OFF_TICKET + (call & 3) * 128);
  if (blockIdx.x == 0 && threadIdx.x == 64)
    __hip_atomic_store((u32*)(p.dst[7] + OFF_TICKET + ((call + 2) & 3) * 128), 0u, __ATOMIC_RELAXED, __HIP_MEMORY_SCOPE_AGENT);
  const size_t pbase = OFF_STAGE + (size_t)(call & 1) * 2 * MAT_BYTES + (ISX ? 0 : MAT_BYTES) + (size_t)p.rank * NCH * SC_BYTES + (size_t)(w4 * 2) * 512 + lane * 16;
  int prev = -1;
  for (int it = 0;; ++it) {
    u32x4 c0;
    if (ISX) {
      c0 = n0;
      asm volatile("" : "+v"(c0));
    } else {
      u32 a0, a1, a2, a3;
      tr4(d0, d1, d2, d3, a0, a1, a2, a3);
      *(u32*)(wp + 0) = a0; *(u32*)(wp + 16) = a1; *(u32*)(wp + 32) = a2; *(u32*)(wp + 48) = a3;
      asm volatile("s_waitcnt lgkmcnt(0)" ::: "memory");
      c0 = *(const u32x4*)(wreg + lane * 16);
      asm volatile("s_waitcnt lgkmcnt(0)" : "+v"(c0) :: "memory");
    }
    const bool hasnext = (unsigned)tnext < (unsigned)NUNIT;
    issue(hasnext ? tnext : tcur);
    if (ISX && hasnext && threadIdx.x == 0)
      asm volatile("global_atomic_add %0, %1, %2, off sc0 sc1" : "=v"(tk) : "v"(tick), "v"(one) : "memory");
    const size_t doff = pbase + (size_t)(tcur >> 5) * SC_BYTES + (size_t)(tcur & 31) * 4096;
#pragma unroll
    for (int i = 0; i < NRANK; ++i) st16_sys(p.dst[i] + doff, c0);
    WAIT_VM(8);
    PIN();
    if (ISX && hasnext && threadIdx.x == 0) tslot[it & 1] = tk + 2 * NPUSH;
    asm volatile("s_waitcnt lgkmcnt(0)" ::: "memory");
    __builtin_amdgcn_s_barrier();
    if (ISX && prev >= 0 && threadIdx.x < NRANK) __hip_atomic_fetch_add(myflag + prev * 32, 1u, __ATOMIC_RELAXED, __HIP_MEMORY_SCOPE_SYSTEM);
    prev = tcur;
    STAMP(1 + (it < 38 ? it : 38));
    if (!hasnext) break;
    tcur = tnext;
    tnext = __builtin_amdgcn_readfirstlane(tslot[it & 1]);
  }
  WAIT_VM(0);
  __builtin_amdgcn_s_barrier();
  if (ISX && threadIdx.x < NRANK) __hip_atomic_fetch_add(myflag + prev * 32, 1u, __ATOMIC_RELAXED, __HIP_MEMORY_SCOPE_SYSTEM);
  STAMP(40);
  if (threadIdx.x == 0) __hip_atomic_store(cnt, call, __ATOMIC_RELAXED, __HIP_MEMORY_SCOPE_AGENT);
}

template <typename T> __device__ __forceinline__ T* uniform_ptr(T* q) {
  unsigned long long v = (unsigned long long)q;
  u32 lo = __builtin_amdgcn_readfirstlane((u32)v), hi = __builtin_amdgcn_readfirstlane((u32)(v >> 32));
  return (T*)(((unsigned long long)hi << 32) | lo);
}
__device__ __forceinline__ void glds1(const u8* gbase, u32 ldsaddr, u32 voff) {
  u32 keep;
  asm volatile("s_mov_b32 %0, m0\n\ts_mov_b32 m0, %2\n\ts_nop 0\n\t"
               "global_load_lds_dwordx4 %1, %3 sc1\n\t"
               "s_mov_b32 m0, %0"
               : "=&s"(keep) : "v"(voff), "s"(ldsaddr), "s"(gbase) : "memory");
}
__device__ __forceinline__ void gemm_stage(const u8* Ag, const u8* Bg, u32 dst, int s, int wid, u32 voff) {
  const int c = s >> 1, s0 = (s & 1) * 4;
  const size_t o0 = (size_t)(s0 * NCH + c) * SC_BYTES + wid * 1024;
  constexpr size_t SS = (size_t)NCH * SC_BYTES;
#pragma unroll
  for (int g = 0; g < 4; ++g) {
    glds1(Ag + o0 + g * SS, dst + g * ASRC + wid * 1024, voff);
    glds1(Bg + o0 + g * SS, dst + 4 * ASRC + g * 8192 + wid * 1024, voff);
  }
  glds1(Ag + (size_t)((s0 + (wid >> 1)) * NCH + c) * SC_BYTES + (8 + (wid & 1)) * 1024, dst + (wid >> 1) * ASRC + (8 + (wid & 1)) * 1024, voff);
}

__device__ __forceinline__ bool flags_ready(const u32* f, u32 target, int lane) {
  u32 v = __hip_atomic_load(f, __ATOMIC_RELAXED, __HIP_MEMORY_SCOPE_SYSTEM);
  return __all(v >= target);
}

__device__ __forceinline__ void gemm_wg(const Params& p, u8* lds, u32* cnt, int g) {
  const int wid = __builtin_amdgcn_readfirstlane(threadIdx.x >> 6), lane = threadIdx.x & 63;
  const u32 call = __builtin_amdgcn_readfirstlane(__hip_atomic_load(cnt, __ATOMIC_RELAXED, __HIP_MEMORY_SCOPE_AGENT)) + 1;
  STAMP(0);
  const int xcd = g & 7, idx = g >> 3;
  const int tl = xcd * (NRB * 2) + idx;
  const int tj = (tl / (NRB * 4)) * 4 + (tl & 3), ti = (tl % (NRB * 4)) >> 2;
  const int row0 = (ti == NRB - 1) ? MM - TM : ti * TM;
  const int col0 = tj * TN;
  u8* ws = p.dst[7];
  const u8* Ag = ws + OFF_STAGE + (size_t)(call & 1) * 2 * MAT_BYTES;
  const u8* Bg = Ag + MAT_BYTES;
  u32* tmo = (u32*)(ws + OFF_TMO);
  const u32 target = call * NRANK;
  const int l7 = lane & 7;
  const int fb = l7 < 2 ? 2 * tj + l7 : (row0 >> 7) + (l7 > 4 ? 2 : l7 - 2);
  const u32* fl = (const u32*)(ws + OFF_FLAG + (size_t)fb * 128);
  bool bad = false;
  float sc0 = p.sx[0] * p.sw[0];
  asm volatile("" : "+v"(sc0));

  auto spin = [&](int c) {
    const u32* f = fl + (size_t)c * 32 * 32;
    unsigned long long t0 = __builtin_amdgcn_s_memrealtime();
    for (u32 it = 1; !flags_ready(f, target, lane); ++it) {
      if ((it & 15) == 0) {
        if (__hip_atomic_load(tmo, __ATOMIC_RELAXED, __HIP_MEMORY_SCOPE_AGENT) == call ||
            __builtin_amdgcn_s_memrealtime() - t0 > DEADLINE_TICKS) {
          __hip_atomic_store(tmo, call, __ATOMIC_RELAXED, __HIP_MEMORY_SCOPE_AGENT);
          bad = true;
          break;
        }
      }
      __builtin_amdgcn_s_sleep(2);
    }
    ACQ();
  };

  const int wm = wid >> 2, wn = wid & 3;
  const int gq = lane >> 4, r = lane & 15;
  const int aoff = gq * ASRC + (wm * MF) * 512 + r * 16;
  const int boff = 4 * ASRC + gq * 8192 + (wn * 4) * 512 + r * 16;
  f32x4 acc[MF][4];
#pragma unroll
  for (int i = 0; i < MF; ++i)
#pragma unroll
    for (int j = 0; j < 4; ++j) acc[i][j] = f32x4{0.f, 0.f, 0.f, 0.f};

  const u32 lds0 = (u32)(size_t)lds;
  const u32 voff = lane * 16;
  const u8* At = uniform_ptr(Ag + (size_t)(row0 >> 4) * 512);
  const u8* Bt = uniform_ptr(Bg + (size_t)(col0 >> 4) * 512);
  spin(0);
  gemm_stage(At, Bt, lds0, 0, wid, voff);
#define LDFRAG(dst, ptr) do { u32x4 lo_ = *(const u32x4*)(ptr); u32x4 hi_ = *(const u32x4*)((ptr) + 256); \
      dst = i32x8{(int)lo_[0], (int)lo_[1], (int)lo_[2], (int)lo_[3], (int)hi_[0], (int)hi_[1], (int)hi_[2], (int)hi_[3]}; } while (0)
  for (int s = 0; s < NSTEP - 1; ++s) {
    const bool need = ((s + 1) & 1) == 0;
    bool ready = true;
    if (need) ready = flags_ready(fl + (size_t)((s + 1) >> 1) * 32 * 32, target, lane);
    WAIT_VM(0);
    __builtin_amdgcn_s_barrier();
    STAMP(1 + s);
    const u8* cur = lds + (s & 1) * STAGE_BYTES;
    const u32 nxt = lds0 + ((s + 1) & 1) * STAGE_BYTES;
    if (ready) {
      if (need) ACQ();
      gemm_stage(At, Bt, nxt, s + 1, wid, voff);
    }
    i32x8 wf[4];
    i32x8 xf[3];
#pragma unroll
    for (int j = 0; j < 4; ++j) LDFRAG(wf[j], cur + boff + j * 512);
    LDFRAG(xf[0], cur + aoff);
    LDFRAG(xf[1], cur + aoff + 512);
    __builtin_amdgcn_sched_barrier(0);
#pragma unroll
    for (int i = 0; i < MF; ++i) {
      if (i + 2 < MF) LDFRAG(xf[(i + 2) % 3], cur + aoff + (i + 2) * 512);
#pragma unroll
      for (int j = 0; j < 4; ++j)
        acc[i][j] = __builtin_amdgcn_mfma_scale_f32_16x16x128_f8f6f4(wf[j], xf[i % 3], acc[i][j], 1, 1, 0, 0x7F7F7F7F, 0, 0x7F7F7F7F);
      __builtin_amdgcn_sched_barrier(0);
    }
#pragma unroll
    for (int i = 0; i < MF; ++i)
      asm volatile("" : "+v"(acc[i][0]), "+v"(acc[i][1]), "+v"(acc[i][2]), "+v"(acc[i][3]));
    if (!ready) {
      spin((s + 1) >> 1);
      gemm_stage(At, Bt, nxt, s + 1, wid, voff);
    }
  }
  {
    WAIT_VM(0);
    __builtin_amdgcn_s_barrier();
    STAMP(NSTEP);
    const u8* cur = lds + ((NSTEP - 1) & 1) * STAGE_BYTES;
    const float sc = bad ? __builtin_nanf("") : sc0;
    const float c2 = -sc * 1.4426950408889634f;
    const float lim = 60.0f * 1.4426950408889634f;
    i32x8 wf[4];
    i32x8 xf[2];
#pragma unroll
    for (int j = 0; j < 4; ++j) LDFRAG(wf[j], cur + boff + j * 512);
    LDFRAG(xf[0], cur + aoff);
#pragma unroll
    for (int i = 0; i <= MF; ++i) {
      if (i < MF) {
        if (i + 1 < MF) LDFRAG(xf[(i + 1) & 1], cur + aoff + (i + 1) * 512);
#pragma unroll
        for (int j = 0; j < 4; ++j)
          acc[i][j] = __builtin_amdgcn_mfma_scale_f32_16x16x128_f8f6f4(wf[j], xf[i & 1], acc[i][j], 1, 1, 0, 0x7F7F7F7F, 0, 0x7F7F7F7F);
      }
      if (i > 0) {
        const int row = row0 + wm * (TM / 2) + (i - 1) * 16 + r;
        {
          float* o = p.out + (size_t)row * NN + col0 + wn * 64 + gq * 4;
#pragma unroll
          for (int j = 0; j < 4; ++j) {
            f32x4 v;
#pragma unroll
            for (int e = 0; e < 4; ++e) {
              const float a = acc[i - 1][j][e];
              const float y = a * sc;
              const float t = __builtin_amdgcn_fmed3f(a * c2, -lim, lim);
              const float ex = __builtin_amdgcn_exp2f(t);
              v[e] = y * __builtin_amdgcn_rcpf(1.0f + ex);
            }
            asm volatile("global_store_dwordx4 %0, %1, off sc0 sc1\n\ts_nop 1" :: "v"(o + j * 16), "v"(v) : "memory");
          }
        }
      }
      __builtin_amdgcn_sched_barrier(0);
    }
  }
  STAMP(40);
  STAMP(41);
  if (threadIdx.x == 0) __hip_atomic_store(cnt, call, __ATOMIC_RELAXED, __HIP_MEMORY_SCOPE_AGENT);
}

__global__ void __launch_bounds__(512, 2) fused_kernel(Params p) {
  extern __shared__ __attribute__((aligned(16))) u8 lds[];
  u32* cnt = (u32*)(p.dst[7] + OFF_WGCNT + (size_t)blockIdx.x * 128);
  if (blockIdx.x >= NPUSH) gemm_wg(p, lds, cnt, blockIdx.x - NPUSH);
  else if (__builtin_amdgcn_readfirstlane(threadIdx.x) < 256) pusher<true>(p, lds, cnt);
  else pusher<false>(p, lds, cnt);
}

extern "C" void kernel_launch(int rank, int nranks, ncclComm_t comm,
                              hipStream_t stream,
                              void* const* d_in, const int* in_sizes, int n_in,
                              void* d_out, int out_size,
                              const int* in_dtypes, int out_dtype,
                              void* const* d_out_all,
                              void* const* d_ws_all, size_t ws_bytes) {
  if (nranks != NRANK || (unsigned)rank >= (unsigned)NRANK || ws_bytes < WS_NEED || n_in < 4) {
    fprintf(stderr, "kernel_launch: unsupported configuration\n");
    return;
  }
  HIP_CHECK(hipFuncSetAttribute((const void*)fused_kernel, hipFuncAttributeMaxDynamicSharedMemorySize, LDS_BYTES));
  Params p;
  p.x = (const u8*)d_in[0]; p.w = (const u8*)d_in[1];
  p.sx = (const float*)d_in[2]; p.sw = (const float*)d_in[3];
  p.out = (float*)d_out;
  for (int i = 0; i < NRANK; ++i) p.dst[i] = (u8*)d_ws_all[(rank + 1 + i) & 7];
  p.rank = rank;
  fused_kernel<<<GRID, 512, LDS_BYTES, stream>>>(p);
  HIP_CHECK(hipGetLastError());
}
```

```cpp
#include <hip/hip_runtime.h>
#include <rccl/rccl.h>
#include <cstdio>

#define HIP_CHECK(expr) do { hipError_t _e = (expr); if (_e != hipSuccess) { \
    fprintf(stderr, "HIP error %s at %s:%d\n", hipGetErrorString(_e), __FILE__, __LINE__); \
    return; } } while (0)

typedef unsigned int u32;
typedef unsigned char u8;
typedef __attribute__((address_space(1))) unsigned int gu32;
typedef __attribute__((ext_vector_type(4))) u32 u32x4;
typedef __attribute__((ext_vector_type(2))) u32 u32x2;
typedef __attribute__((ext_vector_type(8))) int i32x8;
typedef __attribute__((ext_vector_type(4))) float f32x4;

constexpr int MM = 4096, NN = 4096, NRANK = 8, KSH = 512;
constexpr int CK = 32;
constexpr int NCH = KSH / CK;
constexpr int NSTEP = NCH * 2;
constexpr size_t SC_BYTES = 128u << 10;
constexpr int TM = 320, TN = 256, NRB = 13, NCB = 16, MF = TM / 32;
constexpr int NPUSH = 22, GRID = NPUSH + NRB * NCB;
constexpr size_t OFF_WGCNT = 0;
constexpr size_t OFF_TMO = 32u << 10;
constexpr size_t OFF_TICKET = (32u << 10) + 4096;
constexpr size_t OFF_FLAG = 64u << 10;
constexpr size_t OFF_STAGE = 1u << 20;
constexpr size_t MAT_BYTES = 16u << 20;
constexpr size_t WS_NEED = OFF_STAGE + 4 * MAT_BYTES;
constexpr unsigned long long DEADLINE_TICKS = 50000000ull;
constexpr int ASRC = TM * 32;
constexpr int STAGE_BYTES = 4 * ASRC + 32768;
constexpr int LDS_BYTES = 2 * STAGE_BYTES + 64;

struct Params {
  const u8* x; const u8* w; const float* sx; const float* sw; float* out;
  u8* dst[NRANK];
  int rank;
};
__device__ __forceinline__ u8* lane_dst(const Params& p, int l) {
  u8* r = p.dst[0];
#pragma unroll
  for (int i = 1; i < NRANK; ++i) r = ((l & 7) == i) ? p.dst[i] : r;
  return r;
}

#define STAMP(i) do {} while (0)
#define ACQ() __builtin_amdgcn_fence(__ATOMIC_ACQUIRE, "wavefront")
#define WAIT_VM(n) asm volatile("s_waitcnt vmcnt(" #n ")" ::: "memory")

__device__ __forceinline__ void st16_sys(u8* p, u32x4 v) {
  asm volatile("global_store_dwordx4 %0, %1, off sc0 sc1\n\ts_nop 1" :: "v"(p), "v"(v) : "memory");
}

__device__ __forceinline__ void tr4(u32 d0, u32 d1, u32 d2, u32 d3, u32& o0, u32& o1, u32& o2, u32& o3) {
  u32 t0 = __builtin_amdgcn_perm(d1, d0, 0x05010400u), t1 = __builtin_amdgcn_perm(d1, d0, 0x07030602u);
  u32 t2 = __builtin_amdgcn_perm(d3, d2, 0x05010400u), t3 = __builtin_amdgcn_perm(d3, d2, 0x07030602u);
  o0 = __builtin_amdgcn_perm(t2, t0, 0x05040100u); o1 = __builtin_amdgcn_perm(t2, t0, 0x07060302u);
  o2 = __builtin_amdgcn_perm(t3, t1, 0x05040100u); o3 = __builtin_amdgcn_perm(t3, t1, 0x07060302u);
}

#define LD16(dst, addr) asm volatile("global_load_dwordx4 %0, %1, off" : "=v"(dst) : "v"(addr) : "memory")
#define LD4(dst, addr) asm volatile("global_load_dword %0, %1, off" : "=v"(dst) : "v"(addr) : "memory")

constexpr int NUNIT = NCH * 32;
template <bool ISX>
__device__ __forceinline__ void pusher(const Params& p, u8* lds, u32* cnt) {
  const int wid = __builtin_amdgcn_readfirstlane(threadIdx.x >> 6), lane = threadIdx.x & 63, w4 = wid & 3;
  gu32* myflag = (gu32*)(lane_dst(p, lane) + OFF_FLAG) + (p.rank >> 2) * 32;
  asm volatile("" : "+v"(myflag));
  u8* wreg = lds + wid * 1024;
  volatile __attribute__((address_space(3))) u32* tslot = (volatile __attribute__((address_space(3))) u32*)(lds + 8192);
  u32 cv, tk = 0;
  const u32 one = 1;
  asm volatile("global_load_dword %0, %1, off sc1" : "=v"(cv) : "v"(cnt) : "memory");
  u32x4 n0; u32 d0, d1, d2, d3;
  const u8* src = ISX ? p.x + (size_t)(w4 * 32 + (lane >> 5) * 16 + (lane & 15)) * KSH + ((lane >> 4) & 1) * 16
                      : p.w + (size_t)((lane >> 3) * 4) * NN + w4 * 32 + (lane & 7) * 4;
  u8* wp = wreg + ((lane & 7) >> 2) * 512 + (lane >> 5) * 256 + (lane & 3) * 64 + ((lane >> 3) & 3) * 4;
  auto issue = [&](int t) {
    const int c = t >> 5, b = t & 31;
    if (ISX) { LD16(n0, src + (size_t)b * 128 * KSH + c * CK); }
    else { const u8* a = src + (size_t)c * CK * NN + b * 128; LD4(d0, a); LD4(d1, a + NN); LD4(d2, a + 2 * NN); LD4(d3, a + 3 * NN); }
  };
#define PIN() do { if (ISX) asm volatile("" : "+v"(n0), "+v"(tk)); else asm volatile("" : "+v"(d0), "+v"(d1), "+v"(d2), "+v"(d3)); } while (0)
  int tcur = blockIdx.x, tnext = blockIdx.x + NPUSH;
  issue(tcur);
  WAIT_VM(0);
  PIN();
  asm volatile("" : "+v"(cv));
  const u32 call = __builtin_amdgcn_readfirstlane(cv) + 1;
  STAMP(0);
  u32* tick = (u32*)(p.dst[7] + OFF_TICKET + (call & 3) * 128);
  if (blockIdx.x == 0 && threadIdx.x == 64)
    __hip_atomic_store((u32*)(p.dst[7] + OFF_TICKET + ((call + 2) & 3) * 128), 0u, __ATOMIC_RELAXED, __HIP_MEMORY_SCOPE_AGENT);
  const size_t pbase = OFF_STAGE + (size_t)(call & 1) * 2 * MAT_BYTES + (ISX ? 0 : MAT_BYTES) + (size_t)p.rank * NCH * SC_BYTES + (size_t)(w4 * 2) * 512 + lane * 16;
  int prev = -1;
  for (int it = 0;; ++it) {
    u32x4 c0;
    if (ISX) {
      c0 = n0;
      asm volatile("" : "+v"(c0));
    } else {
      u32 a0, a1, a2, a3;
      tr4(d0, d1, d2, d3, a0, a1, a2, a3);
      *(u32*)(wp + 0) = a0; *(u32*)(wp + 16) = a1; *(u32*)(wp + 32) = a2; *(u32*)(wp + 48) = a3;
      asm volatile("s_waitcnt lgkmcnt(0)" ::: "memory");
      c0 = *(const u32x4*)(wreg + lane * 16);
      asm volatile("s_waitcnt lgkmcnt(0)" : "+v"(c0) :: "memory");
    }
    const bool hasnext = (unsigned)tnext < (unsigned)NUNIT;
    issue(hasnext ? tnext : tcur);
    if (ISX && hasnext && threadIdx.x == 0)
      asm volatile("global_atomic_add %0, %1, %2, off sc0 sc1" : "=v"(tk) : "v"(tick), "v"(one) : "memory");
    const size_t doff = pbase + (size_t)(tcur >> 5) * SC_BYTES + (size_t)(tcur & 31) * 4096;
#pragma unroll
    for (int i = 0; i < NRANK; ++i) st16_sys(p.dst[i] + doff, c0);
    WAIT_VM(8);
    PIN();
    if (ISX && hasnext && threadIdx.x == 0) tslot[it & 1] = tk + 2 * NPUSH;
    asm volatile("s_waitcnt lgkmcnt(0)" ::: "memory");
    __builtin_amdgcn_s_barrier();
    if (ISX && prev >= 0 && threadIdx.x < NRANK) __hip_atomic_fetch_add(myflag + prev * 64, 1u, __ATOMIC_RELAXED, __HIP_MEMORY_SCOPE_SYSTEM);
    prev = tcur;
    STAMP(1 + (it < 38 ? it : 38));
    if (!hasnext) break;
    tcur = tnext;
    tnext = __builtin_amdgcn_readfirstlane(tslot[it & 1]);
  }
  WAIT_VM(0);
  __builtin_amdgcn_s_barrier();
  if (ISX && threadIdx.x < NRANK) __hip_atomic_fetch_add(myflag + prev * 64, 1u, __ATOMIC_RELAXED, __HIP_MEMORY_SCOPE_SYSTEM);
  STAMP(40);
  if (threadIdx.x == 0) __hip_atomic_store(cnt, call, __ATOMIC_RELAXED, __HIP_MEMORY_SCOPE_AGENT);
}

template <typename T> __device__ __forceinline__ T* uniform_ptr(T* q) {
  unsigned long long v = (unsigned long long)q;
  u32 lo = __builtin_amdgcn_readfirstlane((u32)v), hi = __builtin_amdgcn_readfirstlane((u32)(v >> 32));
  return (T*)(((unsigned long long)hi << 32) | lo);
}
__device__ __forceinline__ void glds1(const u8* gbase, u32 ldsaddr, u32 voff) {
  u32 keep;
  asm volatile("s_mov_b32 %0, m0\n\ts_mov_b32 m0, %2\n\ts_nop 0\n\t"
               "global_load_lds_dwordx4 %1, %3 sc1\n\t"
               "s_mov_b32 m0, %0"
               : "=&s"(keep) : "v"(voff), "s"(ldsaddr), "s"(gbase) : "memory");
}
__device__ __forceinline__ void gemm_stage(const u8* Ag, const u8* Bg, u32 dst, int s, int wid, u32 voff) {
  const int c = s >> 1, s0 = (s & 1) * 4;
  const size_t o0 = (size_t)(s0 * NCH + c) * SC_BYTES + wid * 1024;
  constexpr size_t SS = (size_t)NCH * SC_BYTES;
#pragma unroll
  for (int g = 0; g < 4; ++g) {
    glds1(Ag + o0 + g * SS, dst + g * ASRC + wid * 1024, voff);
    glds1(Bg + o0 + g * SS, dst + 4 * ASRC + g * 8192 + wid * 1024, voff);
  }
  glds1(Ag + (size_t)((s0 + (wid >> 1)) * NCH + c) * SC_BYTES + (8 + (wid & 1)) * 1024, dst + (wid >> 1) * ASRC + (8 + (wid & 1)) * 1024, voff);
}

__device__ __forceinline__ void gemm_wg(const Params& p, u8* lds, u32* cnt, int g) {
  const int wid = __builtin_amdgcn_readfirstlane(threadIdx.x >> 6), lane = threadIdx.x & 63;
  const u32 call = __builtin_amdgcn_readfirstlane(__hip_atomic_load(cnt, __ATOMIC_RELAXED, __HIP_MEMORY_SCOPE_AGENT)) + 1;
  STAMP(0);
  const int xcd = g & 7, idx = g >> 3;
  const int tl = xcd * (NRB * 2) + idx;
  const int tj = (tl / (NRB * 4)) * 4 + (tl & 3), ti = (tl % (NRB * 4)) >> 2;
  const int row0 = (ti == NRB - 1) ? MM - TM : ti * TM;
  const int col0 = tj * TN;
  u8* ws = p.dst[7];
  const u8* Ag = ws + OFF_STAGE + (size_t)(call & 1) * 2 * MAT_BYTES;
  const u8* Bg = Ag + MAT_BYTES;
  u32* tmo = (u32*)(ws + OFF_TMO);
  const u32 target = call * (NRANK / 2);
  const int l7 = lane & 7, lh = (lane >> 3) & 1;
  const int fb = l7 < 2 ? 2 * tj + l7 : (row0 >> 7) + (l7 > 4 ? 2 : l7 - 2);
  const u32* fl = (const u32*)(ws + OFF_FLAG + (size_t)(fb * 2 + lh) * 128);
  volatile __attribute__((address_space(3))) int* dslot = (volatile __attribute__((address_space(3))) int*)(lds + 2 * STAGE_BYTES);
  float sc0 = p.sx[0] * p.sw[0];
  asm volatile("" : "+v"(sc0));
  int n0 = 0, n1 = 0;
  auto pick = [&]() -> int {
    const int c = lh ? n1 : n0;
    u32 v = __hip_atomic_load(fl + (size_t)(c < NCH ? c : NCH - 1) * 64 * 32, __ATOMIC_RELAXED, __HIP_MEMORY_SCOPE_SYSTEM);
    const unsigned long long m = __ballot(v >= target);
    const bool r0 = n0 < NCH && (m & 0xffull) == 0xffull, r1 = n1 < NCH && (m & 0xff00ull) == 0xff00ull;
    return (r0 && (!r1 || n0 <= n1)) ? 0 : (r1 ? 1 : -1);
  };
  auto pick_wait = [&]() -> int {
    unsigned long long t0 = __builtin_amdgcn_s_memrealtime();
    for (u32 it = 1;; ++it) {
      const int d = pick();
      if (d >= 0) return d;
      if ((it & 15) == 0) {
        if (__hip_atomic_load(tmo, __ATOMIC_RELAXED, __HIP_MEMORY_SCOPE_AGENT) == call ||
            __builtin_amdgcn_s_memrealtime() - t0 > DEADLINE_TICKS) {
          __hip_atomic_store(tmo, call, __ATOMIC_RELAXED, __HIP_MEMORY_SCOPE_AGENT);
          return n0 < NCH ? 2 : 3;
        }
      }
      __builtin_amdgcn_s_sleep(2);
    }
  };
  bool bad = false;
  auto take = [&](int d) -> int {
    if (d >= 2) bad = true;
    ACQ();
    return (d & 1) ? (n1++) * 2 + 1 : (n0++) * 2;
  };

  const int wm = wid >> 2, wn = wid & 3;
  const int gq = lane >> 4, r = lane & 15;
  const int aoff = gq * ASRC + (wm * MF) * 512 + r * 16;
  const int boff = 4 * ASRC + gq * 8192 + (wn * 4) * 512 + r * 16;
  f32x4 acc[MF][4];
#pragma unroll
  for (int i = 0; i < MF; ++i)
#pragma unroll
    for (int j = 0; j < 4; ++j) acc[i][j] = f32x4{0.f, 0.f, 0.f, 0.f};

  const u32 lds0 = (u32)(size_t)lds;
  const u32 voff = lane * 16;
  const u8* At = uniform_ptr(Ag + (size_t)(row0 >> 4) * 512);
  const u8* Bt = uniform_ptr(Bg + (size_t)(col0 >> 4) * 512);
  if (wid == 0) { const int d = pick_wait(); if (lane == 0) dslot[2] = d; }
  asm volatile("s_waitcnt lgkmcnt(0)" ::: "memory");
  __builtin_amdgcn_s_barrier();
  gemm_stage(At, Bt, lds0, take(__builtin_amdgcn_readfirstlane(dslot[2])), wid, voff);
#define LDFRAG(dst, ptr) do { u32x4 lo_ = *(const u32x4*)(ptr); u32x4 hi_ = *(const u32x4*)((ptr) + 256); \
      dst = i32x8{(int)lo_[0], (int)lo_[1], (int)lo_[2], (int)lo_[3], (int)hi_[0], (int)hi_[1], (int)hi_[2], (int)hi_[3]}; } while (0)
  for (int s = 0; s < NSTEP - 1; ++s) {
    if (wid == 0) { const int d = pick(); if (lane == 0) dslot[s & 1] = d; }
    asm volatile("s_waitcnt vmcnt(0) lgkmcnt(0)" ::: "memory");
    __builtin_amdgcn_s_barrier();
    STAMP(1 + s);
    const u8* cur = lds + (s & 1) * STAGE_BYTES;
    const u32 nxt = lds0 + ((s + 1) & 1) * STAGE_BYTES;
    const int dec = __builtin_amdgcn_readfirstlane(dslot[s & 1]);
    if (dec >= 0) gemm_stage(At, Bt, nxt, take(dec), wid, voff);
    i32x8 wf[4];
    i32x8 xf[3];
#pragma unroll
    for (int j = 0; j < 4; ++j) LDFRAG(wf[j], cur + boff + j * 512);
    LDFRAG(xf[0], cur + aoff);
    LDFRAG(xf[1], cur + aoff + 512);
    __builtin_amdgcn_sched_barrier(0);
#pragma unroll
    for (int i = 0; i < MF; ++i) {
      if (i + 2 < MF) LDFRAG(xf[(i + 2) % 3], cur + aoff + (i + 2) * 512);
#pragma unroll
      for (int j = 0; j < 4; ++j)
        acc[i][j] = __builtin_amdgcn_mfma_scale_f32_16x16x128_f8f6f4(wf[j], xf[i % 3], acc[i][j], 1, 1, 0, 0x7F7F7F7F, 0, 0x7F7F7F7F);
      __builtin_amdgcn_sched_barrier(0);
    }
#pragma unroll
    for (int i = 0; i < MF; ++i)
      asm volatile("" : "+v"(acc[i][0]), "+v"(acc[i][1]), "+v"(acc[i][2]), "+v"(acc[i][3]));
    if (dec < 0) {
      if (wid == 0) { const int d = pick_wait(); if (lane == 0) dslot[2] = d; }
      asm volatile("s_waitcnt lgkmcnt(0)" ::: "memory");
      __builtin_amdgcn_s_barrier();
      gemm_stage(At, Bt, nxt, take(__builtin_amdgcn_readfirstlane(dslot[2])), wid, voff);
    }
  }
  {
    WAIT_VM(0);
    __builtin_amdgcn_s_barrier();
    STAMP(NSTEP);
    const u8* cur = lds + ((NSTEP - 1) & 1) * STAGE_BYTES;
    const float sc = bad ? __builtin_nanf("") : sc0;
    const float c2 = -sc * 1.4426950408889634f;
    const float lim = 60.0f * 1.4426950408889634f;
    i32x8 wf[4];
    i32x8 xf[2];
#pragma unroll
    for (int j = 0; j < 4; ++j) LDFRAG(wf[j], cur + boff + j * 512);
    LDFRAG(xf[0], cur + aoff);
#pragma unroll
    for (int i = 0; i <= MF; ++i) {
      if (i < MF) {
        if (i + 1 < MF) LDFRAG(xf[(i + 1) & 1], cur + aoff + (i + 1) * 512);
#pragma unroll
        for (int j = 0; j < 4; ++j)
          acc[i][j] = __builtin_amdgcn_mfma_scale_f32_16x16x128_f8f6f4(wf[j], xf[i & 1], acc[i][j], 1, 1, 0, 0x7F7F7F7F, 0, 0x7F7F7F7F);
      }
      if (i > 0) {
        const int row = row0 + wm * (TM / 2) + (i - 1) * 16 + r;
        {
          float* o = p.out + (size_t)row * NN + col0 + wn * 64 + gq * 4;
#pragma unroll
          for (int j = 0; j < 4; ++j) {
            f32x4 v;
#pragma unroll
            for (int e = 0; e < 4; ++e) {
              const float a = acc[i - 1][j][e];
              const float y = a * sc;
              const float t = __builtin_amdgcn_fmed3f(a * c2, -lim, lim);
              const float ex = __builtin_amdgcn_exp2f(t);
              v[e] = y * __builtin_amdgcn_rcpf(1.0f + ex);
            }
            asm volatile("global_store_dwordx4 %0, %1, off sc0 sc1\n\ts_nop 1" :: "v"(o + j * 16), "v"(v) : "memory");
          }
        }
      }
      __builtin_amdgcn_sched_barrier(0);
    }
  }
  STAMP(40);
  STAMP(41);
  if (threadIdx.x == 0) __hip_atomic_store(cnt, call, __ATOMIC_RELAXED, __HIP_MEMORY_SCOPE_AGENT);
}

__global__ void __launch_bounds__(512, 2) fused_kernel(Params p) {
  extern __shared__ __attribute__((aligned(16))) u8 lds[];
  u32* cnt = (u32*)(p.dst[7] + OFF_WGCNT + (size_t)blockIdx.x * 128);
  if (blockIdx.x >= NPUSH) gemm_wg(p, lds, cnt, blockIdx.x - NPUSH);
  else if (__builtin_amdgcn_readfirstlane(threadIdx.x) < 256) pusher<true>(p, lds, cnt);
  else pusher<false>(p, lds, cnt);
}

extern "C" void kernel_launch(int rank, int nranks, ncclComm_t comm,
                              hipStream_t stream,
                              void* const* d_in, const int* in_sizes, int n_in,
                              void* d_out, int out_size,
                              const int* in_dtypes, int out_dtype,
                              void* const* d_out_all,
                              void* const* d_ws_all, size_t ws_bytes) {
  if (nranks != NRANK || (unsigned)rank >= (unsigned)NRANK || ws_bytes < WS_NEED || n_in < 4) {
    fprintf(stderr, "kernel_launch: unsupported configuration\n");
    return;
  }
  HIP_CHECK(hipFuncSetAttribute((const void*)fused_kernel, hipFuncAttributeMaxDynamicSharedMemorySize, LDS_BYTES));
  Params p;
  p.x = (const u8*)d_in[0]; p.w = (const u8*)d_in[1];
  p.sx = (const float*)d_in[2]; p.sw = (const float*)d_in[3];
  p.out = (float*)d_out;
  for (int i = 0; i < NRANK; ++i) p.dst[i] = (u8*)d_ws_all[(rank + 1 + i) & 7];
  p.rank = rank;
  fused_kernel<<<GRID, 512, LDS_BYTES, stream>>>(p);
  HIP_CHECK(hipGetLastError());
}
```
